```python
import numpy as np

kernel_name = "gpu_gemm_rs_m2048_k2048_n2048_bf16_relu"


ABSMAX_RTOL = 0.008

COMPUTE_DTYPE = "bfloat16"
EPILOGUE = "relu"
IN_QUANT = None
OUT_QUANT = None
IN_DTYPES = {"x": "bfloat16", "w_mat": "bfloat16"}
WORLD_SIZE = 8

def _to_bf16(x):
    u = np.ascontiguousarray(x, dtype=np.float32).view(np.uint32)
    return ((u + 0x7FFF + ((u >> 16) & 1)) & 0xFFFF0000).view(np.float32)

def setup_inputs():

    x = _to_bf16(np.random.randn(2048, 2048).astype(np.float32))
    w_mat = _to_bf16((np.random.randn(2048, 2048) / np.sqrt(2048)).astype(np.float32))
    return {"x": x, "w_mat": w_mat}

def _epilogue(y):
    return np.maximum(y, 0.0).astype(np.float32)

def reference(x, w_mat):
    y = (x @ w_mat).astype(np.float32)
    return _epilogue(y)
```

```cpp
#include <hip/hip_runtime.h>
#include <rccl/rccl.h>
#include <cstdio>
#include <cstdint>

#define HIP_CHECK(expr) do { hipError_t _e = (expr); if (_e != hipSuccess) { \
    fprintf(stderr, "HIP error %s at %s:%d\n", hipGetErrorString(_e), __FILE__, __LINE__); \
    return; } } while (0)

typedef __attribute__((ext_vector_type(8))) __bf16 bf16x8;
typedef __attribute__((ext_vector_type(4))) __bf16 bf16x4;
typedef __attribute__((ext_vector_type(2))) __bf16 bf16x2;
typedef __attribute__((ext_vector_type(2))) float f32x2;
typedef __attribute__((ext_vector_type(4))) float f32x4;
typedef __attribute__((ext_vector_type(4))) unsigned u32x4;
#define LAS __attribute__((address_space(3)))
#define GAS __attribute__((address_space(1)))

constexpr int NR = 8;
constexpr int MDIM = 2048, NDIM = 2048, KLOC = 256, MLOC = 256;
constexpr unsigned SLOT = 1u << 20;
constexpr unsigned BUFB = NR * SLOT;
constexpr size_t CTRL_OFF = 2ull * BUFB;
constexpr size_t WS_NEED = CTRL_OFF + (256u << 10);
constexpr unsigned long long DEADLINE_TICKS = 100000000ull;
constexpr int AUX_SYS = 17;

struct Params {
    const char* x;
    const char* w;
    char* out;
    char* ws[NR];
    int rank;
};

__device__ __forceinline__ unsigned pack2(float a, float b) {
    f32x2 v = {a, b};
    return __builtin_bit_cast(unsigned, __builtin_convertvector(v, bf16x2));
}
__device__ __forceinline__ unsigned nz(unsigned d) { return d ? d : 0x80008000u; }
__device__ __forceinline__ float lo_f(unsigned d) { return __builtin_bit_cast(float, d << 16); }
__device__ __forceinline__ float hi_f(unsigned d) { return __builtin_bit_cast(float, d & 0xffff0000u); }
__device__ __forceinline__ float relu(float y) { return y < 0.f ? 0.f : y; }

__global__ __launch_bounds__(256, 1) void gemm_rs_kernel(Params p) {
    __shared__ __attribute__((aligned(16))) char lds[4 * 16384];
    const int lane = threadIdx.x & 63;
    const int wave = __builtin_amdgcn_readfirstlane(threadIdx.x >> 6);
    const int b = blockIdx.x;
    const int xcd = b & 7, idx = b >> 3;
    const int mj = (xcd >> 1) * 4 + (idx & 3);
    const int nj = (xcd & 1) * 8 + (idx >> 2);
    const int rank = p.rank;
    char* me = p.ws[rank];

    unsigned* cnt = (unsigned*)(me + CTRL_OFF + 4096) + (b * 4 + wave);
    unsigned* tmo = (unsigned*)(me + CTRL_OFF);
    unsigned cval = __hip_atomic_load(cnt, __ATOMIC_RELAXED, __HIP_MEMORY_SCOPE_AGENT);
    unsigned tval = __hip_atomic_load(tmo, __ATOMIC_RELAXED, __HIP_MEMORY_SCOPE_AGENT);

    const int n0 = nj * 128 + wave * 32;
    char* wl = lds + wave * 16384;
    {
        const char* src = p.w + ((size_t)(lane >> 2) * NDIM + n0 + (lane & 3) * 8) * 2;
#pragma unroll
        for (int i = 0; i < 16; ++i)
            __builtin_amdgcn_global_load_lds((const GAS void*)(src + (size_t)i * 16 * NDIM * 2),
                                             (LAS void*)(wl + i * 1024), 16, 0, 0);
    }
    const char* xl = p.x + ((size_t)(mj * 16 + (lane & 15)) * KLOC + (lane >> 4) * 8) * 2;
    auto load_x = [&](bf16x8 (&f)[8], int owner) {
        const char* s = xl + (size_t)owner * MLOC * KLOC * 2;
#pragma unroll
        for (int k = 0; k < 8; ++k) f[k] = *(const bf16x8*)(s + k * 64);
    };
    const int rot = b % 7;
    auto owner_of = [&](int i) {
        int t = rot + i; t = t >= 7 ? t - 7 : t;
        return i == 7 ? rank : ((rank + 1 + t) & 7);
    };
    bf16x8 xa[8], xb[8];
    load_x(xa, owner_of(0));
    load_x(xb, owner_of(1));

    asm volatile("s_waitcnt vmcnt(16)" ::: "memory");
    bf16x8 wf[2][8];
    {
        const int g = lane >> 4, q = (lane >> 2) & 3, pc = lane & 3;
        const char* base = wl + (8 * g + q) * 64 + pc * 16;
#pragma unroll
        for (int s = 0; s < 8; ++s)
#pragma unroll
            for (int t = 0; t < 2; ++t) {
                bf16x4 lo = __builtin_amdgcn_ds_read_tr16_b64_v4bf16((LAS bf16x4*)(base + s * 2048 + t * 8));
                bf16x4 hi = __builtin_amdgcn_ds_read_tr16_b64_v4bf16((LAS bf16x4*)(base + s * 2048 + t * 8 + 256));
                wf[t][s] = __builtin_shufflevector(lo, hi, 0, 1, 2, 3, 4, 5, 6, 7);
            }
    }
    const unsigned call = __builtin_amdgcn_readfirstlane(cval) + 1;
    const unsigned bufoff = (call & 1) * BUFB;
    const unsigned toff = (unsigned)(mj * 16 + nj) * 4096u + wave * 1024u + lane * 16u;
    if (lane == 0) __hip_atomic_store(cnt, call, __ATOMIC_RELAXED, __HIP_MEMORY_SCOPE_AGENT);

    f32x4 acc0, acc1;
    auto mm = [&](const bf16x8 (&f)[8]) {
        acc0 = f32x4{0.f, 0.f, 0.f, 0.f}; acc1 = acc0;
#pragma unroll
        for (int s = 0; s < 8; ++s) {
            acc0 = __builtin_amdgcn_mfma_f32_16x16x32_bf16(wf[0][s], f[s], acc0, 0, 0, 0);
            acc1 = __builtin_amdgcn_mfma_f32_16x16x32_bf16(wf[1][s], f[s], acc1, 0, 0, 0);
        }
    };
    auto push = [&](int owner) {
        u32x4 d = {nz(pack2(acc0[0], acc0[1])), nz(pack2(acc0[2], acc0[3])),
                   nz(pack2(acc1[0], acc1[1])), nz(pack2(acc1[2], acc1[3]))};
        auto rs = __builtin_amdgcn_make_buffer_rsrc(p.ws[owner & 7] + bufoff + (size_t)rank * SLOT, 0, (int)SLOT, 0x00020000);
        __builtin_amdgcn_raw_buffer_store_b128(d, rs, toff, 0, AUX_SYS);
    };
#pragma unroll
    for (int i = 0; i < 8; i += 2) {
        mm(xa);
        if (i + 2 < 8) load_x(xa, owner_of(i + 2));
        push(owner_of(i));
        mm(xb);
        if (i + 3 < 8) load_x(xb, owner_of(i + 3));
        if (i + 1 < 7) push(owner_of(i + 1));
    }

    auto rl = __builtin_amdgcn_make_buffer_rsrc(me + bufoff, 0, (int)BUFB, 0x00020000);
    u32x4 in[7];
    bool bad = __builtin_amdgcn_readfirstlane(tval) != 0;
    {
        const unsigned long long t0 = __builtin_amdgcn_s_memrealtime();
        for (;;) {
            unsigned mn = 0xffffffffu;
#pragma unroll
            for (int i = 0; i < 7; ++i) {
                const int src = (rank + 1 + i) & 7;
                in[i] = __builtin_amdgcn_raw_buffer_load_b128(rl, toff, src * SLOT, AUX_SYS);
            }
#pragma unroll
            for (int i = 0; i < 7; ++i)
                mn = min(mn, min(min(in[i][0], in[i][1]), min(in[i][2], in[i][3])));
            if (__all(mn != 0)) break;
            if (bad || __builtin_amdgcn_readfirstlane(__hip_atomic_load(tmo, __ATOMIC_RELAXED, __HIP_MEMORY_SCOPE_AGENT)) != 0 ||
                __builtin_amdgcn_s_memrealtime() - t0 > DEADLINE_TICKS) {
                if (lane == 0) __hip_atomic_store(tmo, call, __ATOMIC_RELAXED, __HIP_MEMORY_SCOPE_AGENT);
                bad = true;
                break;
            }
            __builtin_amdgcn_s_sleep(8);
        }
    }
    float y[8] = {acc0[0], acc0[1], acc0[2], acc0[3], acc1[0], acc1[1], acc1[2], acc1[3]};
#pragma unroll
    for (int i = 0; i < 7; ++i)
#pragma unroll
        for (int d = 0; d < 4; ++d) {
            y[2 * d] += lo_f(in[i][d]);
            y[2 * d + 1] += hi_f(in[i][d]);
        }
    u32x4 o = {pack2(relu(y[0]), relu(y[1])), pack2(relu(y[2]), relu(y[3])),
               pack2(relu(y[4]), relu(y[5])), pack2(relu(y[6]), relu(y[7]))};
    if (bad) o = u32x4{0x7fc07fc0u, 0x7fc07fc0u, 0x7fc07fc0u, 0x7fc07fc0u};
    *(u32x4*)(p.out + ((size_t)(mj * 16 + (lane & 15)) * NDIM + n0 + (lane >> 4) * 8) * 2) = o;
    const u32x4 z = {0u, 0u, 0u, 0u};
#pragma unroll
    for (int i = 0; i < 7; ++i) {
        const int src = (rank + 1 + i) & 7;
        __builtin_amdgcn_raw_buffer_store_b128(z, rl, toff, src * SLOT, AUX_SYS);
    }
}

extern "C" void kernel_launch(int rank, int nranks, ncclComm_t comm,
                              hipStream_t stream,
                              void* const* d_in, const int* in_sizes, int n_in,
                              void* d_out, int out_size,
                              const int* in_dtypes, int out_dtype,
                              void* const* d_out_all,
                              void* const* d_ws_all, size_t ws_bytes) {
    (void)comm; (void)in_dtypes; (void)out_dtype; (void)d_out_all;
    if (nranks != NR || (unsigned)rank >= (unsigned)NR || n_in < 2 || ws_bytes < WS_NEED ||
        in_sizes[0] != MDIM * KLOC || in_sizes[1] != KLOC * NDIM || out_size != MLOC * NDIM) {
        fprintf(stderr, "kernel_launch: unexpected problem shape\n");
        return;
    }
    Params p;
    p.x = (const char*)d_in[0];
    p.w = (const char*)d_in[1];
    p.out = (char*)d_out;
    for (int i = 0; i < NR; ++i) p.ws[i] = (char*)d_ws_all[i];
    p.rank = rank;
    gemm_rs_kernel<<<256, 256, 0, stream>>>(p);
    HIP_CHECK(hipGetLastError());
}
```

```cpp
#include <hip/hip_runtime.h>
#include <rccl/rccl.h>
#include <cstdio>
#include <cstdint>
#include <cstdlib>

#define HIP_CHECK(expr) do { hipError_t _e = (expr); if (_e != hipSuccess) { \
    fprintf(stderr, "HIP error %s at %s:%d\n", hipGetErrorString(_e), __FILE__, __LINE__); \
    return; } } while (0)

typedef __attribute__((ext_vector_type(8))) __bf16 bf16x8;
typedef __attribute__((ext_vector_type(4))) __bf16 bf16x4;
typedef __attribute__((ext_vector_type(2))) __bf16 bf16x2;
typedef __attribute__((ext_vector_type(2))) float f32x2;
typedef __attribute__((ext_vector_type(4))) float f32x4;
typedef __attribute__((ext_vector_type(4))) unsigned u32x4;
typedef __attribute__((ext_vector_type(2))) unsigned u32x2;
#define LAS __attribute__((address_space(3)))
#define GAS __attribute__((address_space(1)))

constexpr int NR = 8;
constexpr int MDIM = 2048, NDIM = 2048, KLOC = 256, MLOC = 256;
constexpr unsigned POFF = 0;
constexpr unsigned WOFF = 4u << 20;
constexpr unsigned XOFF = 5u << 20;
constexpr unsigned BUFB = (5u << 20) + (512u << 10);
constexpr size_t CTRL_OFF = 3ull * BUFB;
constexpr size_t WS_NEED = CTRL_OFF + (64u << 10);
constexpr unsigned DEADLINE_TICKS = 100000000u;
constexpr int AUX_SYS = 17;
constexpr int L_W = 0, L_XL = 65536, L_XR = 98304, L_FLAG = 131072, L_TOTAL = 131072 + 64;
constexpr unsigned NAN2 = 0x7fc07fc0u;

struct Params {
    const char* x;
    const char* w;
    char* out;
    char* me;
    char* dst[2][4];
    char* crsa[4];
    int rank;
    int na;
    unsigned tz, dz;
    unsigned slead;
    unsigned sub16;
    unsigned pref, topen;
    unsigned rlead, rl2, gmin, gmax;
    unsigned gend, tb0, pad0, pad1;
    unsigned cls[2][4];
    unsigned dt[16];
    unsigned tx[16];
    unsigned tw[32];
    unsigned tl[4];
};

__device__ __forceinline__ unsigned pack2(float a, float b) {
    f32x2 v = {a, b};
    return __builtin_bit_cast(unsigned, __builtin_convertvector(v, bf16x2));
}
__device__ __forceinline__ unsigned nz(unsigned d) { return d | ((d - 1u) & 0x80000000u); }
__device__ __forceinline__ u32x4 nz4(u32x4 d) { return u32x4{nz(d[0]), nz(d[1]), nz(d[2]), nz(d[3])}; }
__device__ __forceinline__ float lo_f(unsigned d) { return __builtin_bit_cast(float, d << 16); }
__device__ __forceinline__ float hi_f(unsigned d) { return __builtin_bit_cast(float, d & 0xffff0000u); }
__device__ __forceinline__ float relu(float y) { return y < 0.f ? 0.f : y; }
__device__ __forceinline__ unsigned min4(u32x4 v) { return min(min(v[0], v[1]), min(v[2], v[3])); }
__device__ __forceinline__ unsigned now32() { return (unsigned)__builtin_amdgcn_s_memrealtime(); }
__device__ __forceinline__ void wait_until(unsigned t0, unsigned d) {
    while (now32() - t0 < d) __builtin_amdgcn_s_sleep(1);
}
__device__ __forceinline__ u32x4 tr_frag(const char* a, int hi_off) {
    u32x2 lo = __builtin_bit_cast(u32x2, __builtin_amdgcn_ds_read_tr16_b64_v4bf16((LAS bf16x4*)a));
    u32x2 hi = __builtin_bit_cast(u32x2, __builtin_amdgcn_ds_read_tr16_b64_v4bf16((LAS bf16x4*)(a + hi_off)));
    return u32x4{lo[0], lo[1], hi[0], hi[1]};
}
#define MFMA(a, bb, c) __builtin_amdgcn_mfma_f32_16x16x32_bf16(__builtin_bit_cast(bf16x8, a), __builtin_bit_cast(bf16x8, bb), c, 0, 0, 0)
#define WAIT_VM0() __builtin_amdgcn_s_waitcnt(0x0F70)
#define WAIT_LGKM0() __builtin_amdgcn_s_waitcnt(0xC07F)
#define SGB(mask, n) __builtin_amdgcn_sched_group_barrier(mask, n, 0)
#define RSRC(base, bytes) __builtin_amdgcn_make_buffer_rsrc((base), 0, (int)(bytes), 0x00020000)

__device__ __forceinline__ void flag_add(char* lds, int f, int lane) {
    const unsigned a = (unsigned)(size_t)(LAS char*)lds + L_FLAG + f * 4, one = 1;
    if (lane == 0) asm volatile("ds_add_u32 %0, %1" :: "v"(a), "v"(one) : "memory");
}
__device__ __forceinline__ void flag_wait(char* lds, int f, unsigned n) {
    const unsigned a = (unsigned)(size_t)(LAS char*)lds + L_FLAG + f * 4;
    for (;;) {
        unsigned v;
        asm volatile("ds_read_b32 %0, %1\n\ts_waitcnt lgkmcnt(0)" : "=&v"(v) : "v"(a) : "memory");
        if ((unsigned)__builtin_amdgcn_readfirstlane(v) >= n) break;
        __builtin_amdgcn_s_sleep(1);
    }
}
enum { F_L1 = 0, F_XR = 1, F_WL = 2  , F_FREE = 6   };

#define STAMP(i) do {} while (0)
#define DBG_FLUSH() do {} while (0)

__global__ __launch_bounds__(512, 2) void gemm_rs_kernel(Params p) {
    __shared__ __attribute__((aligned(16))) char lds[L_TOTAL];
    unsigned long long t064 = __builtin_amdgcn_s_memrealtime();
    const int lane = threadIdx.x & 63;
    const int wave = __builtin_amdgcn_readfirstlane(threadIdx.x >> 6);
    const int b = blockIdx.x;
    {
        const unsigned zero = 0, fa = (unsigned)(size_t)(LAS char*)lds + L_FLAG + (lane & 15) * 4;
        asm volatile("ds_write_b32 %0, %1" :: "v"(fa), "v"(zero) : "memory");
    }
    const int xcd = b & 7, role = (b >> 3) & 1, idx = b >> 4;
    const int u = role == 0 ? idx : (xcd >> 1) + 4 * (idx & 3);
    const int jj = (idx >> 2) * 2 + (xcd & 1);
    const int c = role == 0 ? 2 * idx + (wave & 1) : jj + 8 * (wave >> 1);
    const int ss = (xcd + c) & 7, xt = (2 * xcd + (wave & 1) + idx) & 15;
    const int peer = wave < 2 ? (ss >> 1) : (xt & 3);
    const char* xg = p.x; const char* wg = p.w; char* outp = p.out; char* me = p.me;
    char* pws = p.crsa[peer];
    int rank = p.rank, na = p.na;
    unsigned tz = p.tz, dz = p.dz, slead = p.slead, sub16 = p.sub16, prefm = p.pref, topen = p.topen;
    unsigned txu = p.tx[u], twc = p.tw[c];
    unsigned la0 = p.cls[0][2], la1 = p.cls[0][3], lb0 = p.cls[1][2], lb1 = p.cls[1][3];
    asm volatile("" : "+s"(xg), "+s"(wg), "+s"(outp), "+s"(me), "+s"(pws), "+s"(rank), "+s"(na), "+s"(tz), "+s"(dz),
                      "+s"(slead), "+s"(sub16), "+s"(prefm), "+s"(topen), "+s"(txu), "+s"(twc),
                      "+s"(la0), "+s"(la1), "+s"(lb0), "+s"(lb1), "+s"(t064));
    const unsigned t0 = (unsigned)t064;
    const int g = rank >> 2, gi = rank & 3;
    const bool isA = u < na;
    const int og = isA ? g : 1 - g;
    GAS unsigned* cnt = (GAS unsigned*)(me + CTRL_OFF + 4096) + (b * 8 + wave);
    GAS unsigned* tmo = (GAS unsigned*)(me + CTRL_OFF);
    const unsigned cval = __hip_atomic_load(cnt, __ATOMIC_RELAXED, __HIP_MEMORY_SCOPE_AGENT);
    const int g4 = lane >> 4, q4 = (lane >> 2) & 3, pc = lane & 3;

    if (__builtin_expect(role == 0, 1)) {
        if (__builtin_expect(wave < 4, 1)) {
            u32x4 v[4];
            unsigned dst;
            const unsigned when = wave < 2 ? (twc & 0xffffu) + (((ss & 1) * 2 * sub16) >> 4) : (txu & 0xffffu) + (((xt >> 2) * sub16) >> 4);
            if (when > slead) wait_until(t0, when - slead);
            if (wave < 2) {
                const char* wsrc = wg + ((size_t)(32 * ss + (lane >> 3)) * NDIM + 64 * c + (lane & 7) * 8) * 2;
#pragma unroll
                for (int q = 0; q < 4; ++q) v[q] = *(const GAS u32x4*)(wsrc + (size_t)q * 8 * NDIM * 2);
                char* sl = lds + wave * 4096;
#pragma unroll
                for (int q = 0; q < 4; ++q) *(u32x4*)(sl + q * 1024 + lane * 16) = v[q];
                const char* base = sl + (8 * g4 + q4) * 128 + pc * 16;
                v[0] = tr_frag(base, 512); v[1] = tr_frag(base + 8, 512);
                v[2] = tr_frag(base + 64, 512); v[3] = tr_frag(base + 72, 512);
                dst = WOFF + (unsigned)(gi * 32 + c) * 8192u + (ss & 1) * 2048u + lane * 16u;
            } else {
                const int xo = xt >> 2;
                const char* xs = xg + ((size_t)((4 * (1 - og) + xo) * MLOC + 16 * u + (lane & 15)) * KLOC + peer * 64 + g4 * 8) * 2;
                v[0] = *(const GAS u32x4*)xs;
                v[1] = *(const GAS u32x4*)(xs + 64);
                v[2] = v[0]; v[3] = v[1];
                dst = XOFF + (unsigned)((gi * 16 + u) * 4 + xo) * 2048u + lane * 16u;
            }
#pragma unroll
            for (int q = 0; q < 4; ++q) v[q] = nz4(v[q]);
            const unsigned call = __builtin_amdgcn_readfirstlane(cval) + 1;
            auto rp = RSRC(pws + (call % 3) * BUFB, BUFB);
            STAMP(1);
            wait_until(t0, when);
            __builtin_amdgcn_raw_buffer_store_b128(v[0], rp, dst, 0, AUX_SYS);
            __builtin_amdgcn_raw_buffer_store_b128(v[1], rp, dst + 1024, 0, AUX_SYS);
            if (wave < 2) {
                __builtin_amdgcn_raw_buffer_store_b128(v[2], rp, dst + 4096, 0, AUX_SYS);
                __builtin_amdgcn_raw_buffer_store_b128(v[3], rp, dst + 4096 + 1024, 0, AUX_SYS);
            }
            if (lane == 0) __hip_atomic_store(cnt, call, __ATOMIC_RELAXED, __HIP_MEMORY_SCOPE_AGENT);
            STAMP(2);
        } else {
            const unsigned call = __builtin_amdgcn_readfirstlane(cval) + 1;
            if (lane == 0) __hip_atomic_store(cnt, call, __ATOMIC_RELAXED, __HIP_MEMORY_SCOPE_AGENT);
            auto rz = RSRC(me + ((call + 2) % 3) * BUFB, BUFB);
            const u32x4 z = {0u, 0u, 0u, 0u};
            const unsigned zo = (unsigned)((idx * 8 + xcd) * 4 + (wave - 4)) * 11264u + lane * 16u;
            const unsigned zph = ((unsigned)(idx * 4 + (wave - 4)) * dz) >> 6;
#pragma unroll 1
            for (int q = 0; q < 11; ++q) {
                wait_until(t0, tz + zph + q * dz);
                __builtin_amdgcn_raw_buffer_store_b128(z, rz, zo + q * 1024u, 0, AUX_SYS);
            }
        }
        DBG_FLUSH();
        return;
    }

    asm volatile("s_waitcnt lgkmcnt(0)\n\ts_barrier" ::: "memory");
    const int grp = wave >> 1, w2 = wave & 1;
    const unsigned tval = __hip_atomic_load(tmo, __ATOMIC_RELAXED, __HIP_MEMORY_SCOPE_AGENT);
    const size_t wlane = ((size_t)(lane >> 3) * NDIM + ((lane & 7) ^ (((lane >> 4) & 1) << 2)) * 8) * 2;
    auto load_w = [&](int cg, int buf) {
        const char* wsrc = wg + wlane + ((size_t)(128 * w2) * NDIM + 64 * cg) * 2;
#pragma unroll
        for (int i = 0; i < 16; ++i)
            __builtin_amdgcn_global_load_lds((const GAS void*)(wsrc + (size_t)i * 8 * NDIM * 2),
                                             (LAS void*)(lds + L_W + buf * 32768 + w2 * 16384 + i * 1024), 16, 0, 0);
    };
    const unsigned tp0 = isA ? la0 + jj * la1 : lb0 + jj * lb1;
    if (grp == 1) {
        if (tp0) wait_until(t0, tp0);
        load_w(jj, 0);
#pragma unroll
        for (int q = 0; q < 4; ++q) {
            const char* xsrc = xg + ((size_t)((4 * og + ((gi + q) & 3)) * MLOC + 16 * u + (lane >> 3)) * KLOC + ((lane & 7) ^ (lane >> 3)) * 8) * 2 + w2 * 256;
#pragma unroll
            for (int i = 0; i < 4; ++i)
                __builtin_amdgcn_global_load_lds((const GAS void*)(xsrc + (i >> 1) * 128 + (i & 1) * 8 * KLOC * 2),
                                                 (LAS void*)(lds + L_XL + q * 8192 + w2 * 4096 + i * 1024), 16, 0, 0);
        }
    }
    const int ci = isA ? 0 : 1;
    char* dst0 = p.dst[ci][0]; char* dst1 = p.dst[ci][1]; char* dst2 = p.dst[ci][2]; char* dst3 = p.dst[ci][3];
    unsigned rlead = p.rlead, rl2 = p.rl2, gmin = p.gmin, gmax = p.gmax, gend = p.gend, tb0 = p.tb0;
    unsigned r0 = p.cls[ci][0], rd = p.cls[ci][1], dtx = p.dt[ci * 8 + xcd], tlc = p.tl[grp], tl0 = p.tl[0], tl1 = p.tl[1];
    asm volatile("" : "+s"(dst0), "+s"(dst1), "+s"(dst2), "+s"(dst3), "+s"(rlead), "+s"(rl2), "+s"(gmin), "+s"(gmax),
                      "+s"(gend), "+s"(tb0), "+s"(r0), "+s"(rd), "+s"(dtx), "+s"(tlc), "+s"(tl0), "+s"(tl1));
    txu >>= 16;
    twc >>= 16;
    const char* xl_rd0 = lds + L_XL + (lane & 15) * 128 + (g4 ^ (lane & 7)) * 16;
    const char* xl_rd1 = lds + L_XL + (lane & 15) * 128 + ((4 + g4) ^ (lane & 7)) * 16;
    const char* w_rd = lds + L_W + (grp & 1) * 32768 + (8 * g4 + q4) * 128 + ((w2 ^ (q4 >> 1)) * 64) + pc * 16;
    const unsigned wroff = WOFF + (unsigned)(c * 2 + w2) * 4096u + lane * 16u;
    const unsigned toff = POFF + (unsigned)((((c >> 1) * 16 + u) * 2 + (c & 1)) * 2 + w2) * 1024u + lane * 16u;
    WAIT_VM0();
    if (grp == 1) flag_add(lds, F_L1, lane);
    const unsigned call = __builtin_amdgcn_readfirstlane(cval) + 1;
    const unsigned bufoff = (call % 3) * BUFB;
    bool bad = __builtin_amdgcn_readfirstlane(tval) != 0;
    if (lane == 0) __hip_atomic_store(cnt, call, __ATOMIC_RELAXED, __HIP_MEMORY_SCOPE_AGENT);
    auto rl = RSRC(me + bufoff, BUFB);
    auto give_up = [&]() -> bool {
        const unsigned el = now32() - t0;
        if (!bad && el < 10000u) return false;
        if (bad || el > DEADLINE_TICKS ||
            __builtin_amdgcn_readfirstlane(__hip_atomic_load(tmo, __ATOMIC_RELAXED, __HIP_MEMORY_SCOPE_AGENT)) != 0) {
            if (lane == 0) __hip_atomic_store(tmo, call, __ATOMIC_RELAXED, __HIP_MEMORY_SCOPE_AGENT);
            bad = true;
            return true;
        }
        __builtin_amdgcn_s_sleep(64);
        return false;
    };
    if (grp >= 2) {
        const int src = wave - 4;
        auto load_w1 = [&]() {
            wait_until(t0, tp0 + tl1);
            const char* wsrc = wg + wlane + ((size_t)(64 * src) * NDIM + 64 * (jj + 8)) * 2;
#pragma unroll
            for (int i = 0; i < 8; ++i)
                __builtin_amdgcn_global_load_lds((const GAS void*)(wsrc + (size_t)i * 8 * NDIM * 2),
                                                 (LAS void*)(lds + L_W + 32768 + src * 8192 + i * 1024), 16, 0, 0);
            WAIT_VM0();
            flag_add(lds, F_WL + 1, lane);
        };
        const bool xfirst = txu <= tp0 + tl0;
        if (!xfirst) load_w1();
        wait_until(t0, txu);
        u32x4 xr[8];
        for (;;) {
            unsigned mn = 0xffffffffu;
#pragma unroll
            for (int i = 0; i < 8; ++i)
                xr[i] = __builtin_amdgcn_raw_buffer_load_b128(rl, XOFF + (unsigned)(src * 16 + u) * 8192u + i * 1024u + lane * 16u, 0, AUX_SYS);
#pragma unroll
            for (int i = 0; i < 8; ++i) mn = min(mn, min4(xr[i]));
            if (__all(mn != 0)) break;
            if (give_up()) {
#pragma unroll
                for (int i = 0; i < 8; ++i) xr[i] = u32x4{NAN2, NAN2, NAN2, NAN2};
                break;
            }
        }
#pragma unroll
        for (int i = 0; i < 8; ++i) *(u32x4*)(lds + L_XR + src * 8192 + i * 1024 + lane * 16) = xr[i];
        WAIT_LGKM0();
        flag_add(lds, F_XR, lane);
        STAMP(3);
        if (xfirst) load_w1();
        flag_wait(lds, F_FREE + (grp & 1), 2);
        wait_until(t0, tlc);
        load_w(c, grp & 1);
        WAIT_VM0();
        flag_add(lds, F_WL + grp, lane);
    }
    u32x4 wr[2][8];
    auto load_wr = [&](unsigned bytes) {
        auto rr = RSRC(me + bufoff, bytes);
#pragma unroll
        for (int k = 0; k < 8; ++k) {
            wr[0][k] = __builtin_amdgcn_raw_buffer_load_b128(rr, wroff + ((k & 1) * 2) * 1024u, (k >> 1) * (32 * 8192), AUX_SYS);
            wr[1][k] = __builtin_amdgcn_raw_buffer_load_b128(rr, wroff + ((k & 1) * 2 + 1) * 1024u, (k >> 1) * (32 * 8192), AUX_SYS);
        }
    };
    const bool pref = twc <= max(prefm, now32() - t0 + 25u);
    if (pref) wait_until(t0, twc);
    load_wr(pref ? BUFB : 0u);
    if (grp >= 1) flag_wait(lds, F_WL + grp, grp == 1 ? 4 : 2);
    flag_wait(lds, F_L1, 2);
    STAMP(1);

    f32x4 acc[4][2];
    {
#pragma unroll
        for (int q = 0; q < 4; ++q) { acc[q][0] = f32x4{0.f, 0.f, 0.f, 0.f}; acc[q][1] = acc[q][0]; }
        u32x4 wf[2][2], xf[2][4];
        auto rdl = [&](int s) {
            wf[s & 1][0] = tr_frag(w_rd + s * 4096, 512);
            xf[s & 1][0] = *(const u32x4*)(((s & 1) ? xl_rd1 : xl_rd0) + (0 * 4 + (s >> 1)) * 2048);
            wf[s & 1][1] = tr_frag(w_rd + s * 4096 + 8, 512);
#pragma unroll
            for (int q = 1; q < 4; ++q)
                xf[s & 1][q] = *(const u32x4*)(((s & 1) ? xl_rd1 : xl_rd0) + (q * 4 + (s >> 1)) * 2048);
        };
        rdl(0);
        __builtin_amdgcn_sched_barrier(0);
#pragma unroll
        for (int s = 0; s < 8; ++s) {
            if (s + 1 < 8) rdl(s + 1);
#pragma unroll
            for (int q = 0; q < 4; ++q) {
                acc[q][0] = MFMA(wf[s & 1][0], xf[s & 1][q], acc[q][0]);
                acc[q][1] = MFMA(wf[s & 1][1], xf[s & 1][q], acc[q][1]);
            }
            if (s < 7) {
                SGB(0x100, 2); SGB(0x8, 1); SGB(0x100, 2); SGB(0x8, 1); SGB(0x100, 1); SGB(0x8, 1); SGB(0x100, 1); SGB(0x8, 1);
                SGB(0x100, 1); SGB(0x8, 1); SGB(0x100, 1); SGB(0x8, 3);
            }
            __builtin_amdgcn_sched_barrier(0);
        }
        if (grp < 2) flag_add(lds, F_FREE + grp, lane);
    }
    STAMP(2);
    for (;;) {
        unsigned mn = 0xffffffffu;
#pragma unroll
        for (int k = 0; k < 8; ++k) mn = min(mn, min(min4(wr[0][k]), min4(wr[1][k])));
        if (__all(mn != 0)) break;
        if (give_up()) { wr[0][0] = u32x4{NAN2, NAN2, NAN2, NAN2}; wr[1][0] = wr[0][0]; break; }
        wait_until(t0, twc);
        load_wr(BUFB);
    }
    STAMP(4);
    const unsigned tres = r0 + (((unsigned)(isA ? (c & ~1) * na + 2 * u : (c & ~1) * (16 - na) + 2 * (u - na)) * rd) >> 4);
    unsigned xr_a[4];
#pragma unroll
    for (int q = 0; q < 4; ++q) { xr_a[q] = (unsigned)(size_t)(LAS char*)lds + L_XR + ((gi + q) & 3) * 2048 + lane * 16; asm volatile("" : "+v"(xr_a[q])); }
    flag_wait(lds, F_XR, 4);
    u32x4 xf[2][8], d[4];
#pragma unroll
    for (int s = 0; s < 8; ++s) xf[0][s] = *(const LAS u32x4*)(size_t)(xr_a[1] + (s >> 1) * 8192 + (s & 1) * 1024);
    unsigned long long slot = 0;
    auto reserve = [&](unsigned long long want) {
        unsigned vz = 0;
        asm volatile("" : "+v"(vz));
        GAS unsigned long long* pacer = (GAS unsigned long long*)(me + CTRL_OFF + 1024 + (ci * 8 + xcd) * 128 + vz);
        if (!isA && want < t064 + tb0) want = t064 + tb0;
        if (lane == 0) {
            __hip_atomic_fetch_max(pacer, want << 4, __ATOMIC_RELAXED, __HIP_MEMORY_SCOPE_WORKGROUP);
            slot = __hip_atomic_fetch_add(pacer, (unsigned long long)dtx, __ATOMIC_RELAXED, __HIP_MEMORY_SCOPE_WORKGROUP);
        }
    };
    const unsigned long long tn = __builtin_amdgcn_s_memrealtime();
    const unsigned tn4 = (unsigned)tn - t0;
    const bool open = isA && tn4 < topen;
    const bool early = open || tn4 + 55u >= tres;
    if (early) reserve(tn + rlead - 100u);
    STAMP(5);
    unsigned gate;
    {
        const unsigned poff = toff + ((unsigned)gi << 20);
        auto pack1 = [&](int q, int k) { d[q][k] = nz(pack2(acc[q][k >> 1][(2 * k) & 3], acc[q][k >> 1][(2 * k + 1) & 3])); };
        auto push = [&](int q, unsigned bytes) {
            auto rs = RSRC((q == 0 ? dst0 : q == 1 ? dst1 : q == 2 ? dst2 : dst3) + bufoff, bytes);
            __builtin_amdgcn_raw_buffer_store_b128(d[q], rs, poff, 0, AUX_SYS);
        };
        const unsigned ob = open ? BUFB : 0u;
        __builtin_amdgcn_sched_barrier(0);
#pragma unroll
        for (int qq = 0; qq < 4; ++qq) {
            const int q = (qq + 1) & 3;
#pragma unroll
            for (int s = 0; s < 8; ++s) {
                acc[q][0] = MFMA(wr[0][s], xf[qq & 1][s], acc[q][0]);
                if (qq + 1 < 4) xf[(qq + 1) & 1][s] = *(const LAS u32x4*)(size_t)(xr_a[(qq + 2) & 3] + (s >> 1) * 8192 + (s & 1) * 1024);
                __builtin_amdgcn_sched_barrier(0);
                acc[q][1] = MFMA(wr[1][s], xf[qq & 1][s], acc[q][1]);
                if (qq > 0 && s >= 1 && s <= 4) pack1(qq, s - 1);
                if (qq > 0 && s == 5) push(qq, ob);
                __builtin_amdgcn_sched_barrier(0);
            }
        }
        if (!isA) { pack1(0, 0); pack1(0, 1); pack1(0, 2); pack1(0, 3); }
        asm volatile("" : "+v"(slot));
        gate = __builtin_amdgcn_readfirstlane((unsigned)(slot >> 4)) - t0;
        STAMP(6);
        if (open) {
            gate = tn4 + 50u;
        } else {
            if (!early) {
                wait_until(t0, tres);
                reserve(__builtin_amdgcn_s_memrealtime() + rl2 - 100u);
                gate = __builtin_amdgcn_readfirstlane((unsigned)(slot >> 4)) - t0;
            }
            if (gate > 4000u) gate = 0;
            wait_until(t0, gate);
            push(1, BUFB); push(2, BUFB); push(3, BUFB);
            if (!isA) push(0, BUFB);
        }
    }
    STAMP(7);
    const unsigned so0 = __builtin_amdgcn_readfirstlane((unsigned)((gi + (isA ? 1 : 0)) & 3) << 20), so1 = (unsigned)((gi + 1) & 3) << 20,
                   so2 = (unsigned)((gi + 2) & 3) << 20, so3 = (unsigned)((gi + 3) & 3) << 20;
    GAS u32x4* optr = (GAS u32x4*)(outp + ((size_t)(u * 16 + (lane & 15)) * NDIM + c * 64 + w2 * 32 + g4 * 8) * 2);
    u32x4 in[4];
    wait_until(t0, max(gate + gmin, min(gate + gmax, gend)));
    for (;;) {
        in[0] = __builtin_amdgcn_raw_buffer_load_b128(rl, toff, so0, AUX_SYS);
        in[1] = __builtin_amdgcn_raw_buffer_load_b128(rl, toff, so1, AUX_SYS);
        in[2] = __builtin_amdgcn_raw_buffer_load_b128(rl, toff, so2, AUX_SYS);
        in[3] = __builtin_amdgcn_raw_buffer_load_b128(rl, toff, so3, AUX_SYS);
        if (__all(min(min(min4(in[0]), min4(in[1])), min(min4(in[2]), min4(in[3]))) != 0)) break;
        if (give_up()) break;
    }
    STAMP(9);
    float y[8];
#pragma unroll
    for (int k = 0; k < 4; ++k) {
        y[2 * k] = isA ? acc[0][k >> 1][(2 * k) & 3] : lo_f(in[0][k]);
        y[2 * k + 1] = isA ? acc[0][k >> 1][(2 * k + 1) & 3] : hi_f(in[0][k]);
    }
#pragma unroll
    for (int q = 1; q < 4; ++q)
#pragma unroll
        for (int k = 0; k < 4; ++k) {
            y[2 * k] += lo_f(in[q][k]);
            y[2 * k + 1] += hi_f(in[q][k]);
        }
    u32x4 o = {pack2(relu(y[0]), relu(y[1])), pack2(relu(y[2]), relu(y[3])),
               pack2(relu(y[4]), relu(y[5])), pack2(relu(y[6]), relu(y[7]))};
    if (bad) o = u32x4{NAN2, NAN2, NAN2, NAN2};
    *optr = o;
}

extern "C" void kernel_launch(int rank, int nranks, ncclComm_t comm,
                              hipStream_t stream,
                              void* const* d_in, const int* in_sizes, int n_in,
                              void* d_out, int out_size,
                              const int* in_dtypes, int out_dtype,
                              void* const* d_out_all,
                              void* const* d_ws_all, size_t ws_bytes) {
    (void)comm; (void)in_dtypes; (void)out_dtype; (void)d_out_all;
    if (nranks != NR || (unsigned)rank >= (unsigned)NR || n_in < 2 || ws_bytes < WS_NEED ||
        in_sizes[0] != MDIM * KLOC || in_sizes[1] != KLOC * NDIM || out_size != MLOC * NDIM) {
        fprintf(stderr, "kernel_launch: unexpected problem shape\n");
        return;
    }
    Params p = {};
    p.x = (const char*)d_in[0];
    p.w = (const char*)d_in[1];
    p.out = (char*)d_out;
    p.me = (char*)d_ws_all[rank];
    for (int q = 0; q < 4; ++q) {
        p.dst[0][q] = (char*)d_ws_all[(rank & 4) + ((rank + q) & 3)];
        p.dst[1][q] = (char*)d_ws_all[((rank & 4) ^ 4) + ((rank + q) & 3)];
        p.crsa[q] = (char*)d_ws_all[((rank & 4) ^ 4) + q];
    }
    p.rank = rank;
    p.na = 10;
    unsigned t_first = 95, unit16 = 220;
    const char* order = "XXWWXXWWXWXWXWXWXWXWWWXWXWXWXWXWXWWWWWWWWWWWWWWWW";
    p.tz = 200; p.dz = 110; p.slead = 130; p.pref = 220; p.topen = 300;
    p.rlead = 155; p.rl2 = 130; p.gmin = 50; p.gmax = 250; p.gend = 1480; p.tb0 = 760;
    p.tl[0] = 200; p.tl[1] = 95; p.tl[2] = 400; p.tl[3] = 460;
    unsigned da = 56, db = 56, lead = 40, leadmax = 110, tlb = 150, tlj = 10, nfast = 10, ra0 = 200, rb0 = 660;
    p.sub16 = unit16 / 4;
    p.cls[0][0] = ra0; p.cls[0][1] = da; p.cls[0][2] = 0; p.cls[0][3] = tlj;
    p.cls[1][0] = rb0; p.cls[1][1] = db; p.cls[1][2] = tlb; p.cls[1][3] = 0;
    int nx = 0, nw = 0; unsigned pos = 0;
    for (const char* s = order; *s; ++s) {
        if (*s == 'X' && nx < 16) { const unsigned t = t_first + ((pos++ * unit16) >> 4); p.tx[nx] = t | ((t + (nx < p.na ? lead : leadmax)) << 16); ++nx; }
        else if (*s == 'W' && nw < 32) { const unsigned t = t_first + ((pos++ * unit16) >> 4); p.tw[nw] = t | ((t + (nw < (int)nfast ? lead : leadmax)) << 16); ++nw; }
    }
    if (nx != 16 || nw != 32) { fprintf(stderr, "bad order\n"); return; }
    for (int x = 0; x < 8; ++x) {
        int ra = 0, rb = 0;
        for (int k = 0; k < 4; ++k) (((x >> 1) + 4 * k) < p.na ? ra : rb)++;
        p.dt[x] = ra ? da * p.na / ra : 0;
        p.dt[8 + x] = rb ? db * (16 - p.na) / rb : 0;
    }
    gemm_rs_kernel<<<256, 512, 0, stream>>>(p);
    HIP_CHECK(hipGetLastError());
}
```

```cpp
#include <hip/hip_runtime.h>
#include <rccl/rccl.h>
#include <cstdio>
#include <cstdint>
#include <cstdlib>

#define HIP_CHECK(expr) do { hipError_t _e = (expr); if (_e != hipSuccess) { \
    fprintf(stderr, "HIP error %s at %s:%d\n", hipGetErrorString(_e), __FILE__, __LINE__); \
    return; } } while (0)

typedef __attribute__((ext_vector_type(8))) __bf16 bf16x8;
typedef __attribute__((ext_vector_type(4))) __bf16 bf16x4;
typedef __attribute__((ext_vector_type(2))) __bf16 bf16x2;
typedef __attribute__((ext_vector_type(2))) float f32x2;
typedef __attribute__((ext_vector_type(4))) float f32x4;
typedef __attribute__((ext_vector_type(4))) unsigned u32x4;
typedef __attribute__((ext_vector_type(2))) unsigned u32x2;
#define LAS __attribute__((address_space(3)))
#define GAS __attribute__((address_space(1)))

#define NA 9
#define NSPLIT 4
#define XSL 0
#define NEARLY 4

constexpr int NR = 8;
constexpr int MDIM = 2048, NDIM = 2048, KLOC = 256, MLOC = 256;
constexpr unsigned POFF = 0;
constexpr unsigned IOFF = 4u << 20;
constexpr unsigned XIN = 32u * 8192u;
constexpr unsigned ISRC = XIN + 17u * 8192u;
constexpr unsigned BUFB = IOFF + 4u * ISRC;
constexpr size_t CTRL_OFF = 3ull * BUFB;
constexpr size_t WS_NEED = CTRL_OFF + (64u << 10);
constexpr unsigned DEADLINE_TICKS = 100000000u;
constexpr int AUX_SYS = 17;
constexpr int L_W = 0, L_XL = 65536, L_XR = 98304, L_FLAG = 131072, L_TOTAL = 131072 + 64;
constexpr unsigned NAN2 = 0x7fc07fc0u;

struct Params {
    const char* w;
    char* me;
    const char* xg[2];
    const char* xl[2][4];
    char* dst[2][4];
    char* crs[4];
    char* out;
    unsigned tz, dz;
    unsigned slead, sub16, pref, rlead, rblk, rl2, gmin, gmax;
    unsigned gend, tb0, sprint, sprc;
    unsigned cls[2][4];
    unsigned dt[16];
    unsigned tx[20];
    unsigned tw[32];
    unsigned tl[4];
};

__device__ __forceinline__ unsigned pack2(float a, float b) {
    f32x2 v = {a, b};
    return __builtin_bit_cast(unsigned, __builtin_convertvector(v, bf16x2));
}
__device__ __forceinline__ unsigned nz(unsigned d) { return d | ((d - 1u) & 0x80000000u); }
__device__ __forceinline__ u32x4 nz4(u32x4 d) { return u32x4{nz(d[0]), nz(d[1]), nz(d[2]), nz(d[3])}; }
__device__ __forceinline__ float lo_f(unsigned d) { return __builtin_bit_cast(float, d << 16); }
__device__ __forceinline__ float hi_f(unsigned d) { return __builtin_bit_cast(float, d & 0xffff0000u); }
__device__ __forceinline__ float relu(float y) { return y < 0.f ? 0.f : y; }
__device__ __forceinline__ unsigned min4(u32x4 v) { return min(min(v[0], v[1]), min(v[2], v[3])); }
__device__ __forceinline__ unsigned now32() { return (unsigned)__builtin_amdgcn_s_memrealtime(); }
__device__ __forceinline__ void wait_until(unsigned t0, unsigned d) {
    while (now32() - t0 < d) __builtin_amdgcn_s_sleep(1);
}
__device__ __forceinline__ u32x4 tr_frag(const char* a, int hi_off) {
    u32x2 lo = __builtin_bit_cast(u32x2, __builtin_amdgcn_ds_read_tr16_b64_v4bf16((LAS bf16x4*)a));
    u32x2 hi = __builtin_bit_cast(u32x2, __builtin_amdgcn_ds_read_tr16_b64_v4bf16((LAS bf16x4*)(a + hi_off)));
    return u32x4{lo[0], lo[1], hi[0], hi[1]};
}
#define MFMA(a, bb, c) __builtin_amdgcn_mfma_f32_16x16x32_bf16(__builtin_bit_cast(bf16x8, a), __builtin_bit_cast(bf16x8, bb), c, 0, 0, 0)
#define WAIT_VM0() __builtin_amdgcn_s_waitcnt(0x0F70)
#define WAIT_LGKM0() __builtin_amdgcn_s_waitcnt(0xC07F)
#define SGB(mask, n) __builtin_amdgcn_sched_group_barrier(mask, n, 0)
#define SB() __builtin_amdgcn_sched_barrier(0)
#define RSRC(base, bytes) __builtin_amdgcn_make_buffer_rsrc((base), 0, (int)(bytes), 0x00020000)
#define GLDS(src, dstlds) __builtin_amdgcn_global_load_lds((const GAS void*)(src), (LAS void*)(dstlds), 16, 0, 0)

__device__ __forceinline__ void flag_add(char* lds, int f, int lane) {
    const unsigned a = (unsigned)(size_t)(LAS char*)lds + L_FLAG + f * 4, one = 1;
    if (lane == 0) asm volatile("ds_add_u32 %0, %1" :: "v"(a), "v"(one) : "memory");
}
__device__ __forceinline__ void flag_wait(char* lds, int f, unsigned n) {
    const unsigned a = (unsigned)(size_t)(LAS char*)lds + L_FLAG + f * 4;
    for (;;) {
        unsigned v;
        asm volatile("ds_read_b32 %0, %1\n\ts_waitcnt lgkmcnt(0)" : "=&v"(v) : "v"(a) : "memory");
        if ((unsigned)__builtin_amdgcn_readfirstlane(v) >= n) break;
        __builtin_amdgcn_s_sleep(1);
    }
}
enum { F_XR = 0  , F_LA = 4, F_LB = 5, F_WL = 6  , F_FREE = 10   };

#define STAMP(i) do {} while (0)
#define DBG_FLUSH() do {} while (0)

__global__ __launch_bounds__(512, 2) void gemm_rs_kernel(Params p) {
    __shared__ __attribute__((aligned(16))) char lds[L_TOTAL];
    const char* wg = p.w; char* me = p.me; const char* xg0 = p.xg[0]; const char* xg1 = p.xg[1];
    const unsigned long long t064 = __builtin_amdgcn_s_memrealtime();
    const unsigned t0 = (unsigned)t064;
    const unsigned lane = threadIdx.x & 63;
    const unsigned wave = __builtin_amdgcn_readfirstlane(threadIdx.x >> 6);
    const unsigned b = blockIdx.x;
    const unsigned xcd = b & 7, idx = b >> 4;
    const unsigned g4 = lane >> 4, q4 = (lane >> 2) & 3, pc = lane & 3;

    if (__builtin_expect((b & 8) == 0, 1)) {
        const bool isW = wave < 2;
        const unsigned c = 2 * idx + (wave & 1), ss = (xcd + c) & 7, xt = (2 * xcd + (wave & 1) + idx) & 15;
        const unsigned peer = isW ? ss >> 1 : xt & 3, xo = xt >> 2;
        const unsigned xslot = wave < 4 ? idx : 16, u = wave < 4 ? idx : NA;
        const bool xa = idx < NA || wave >= 4;
        const unsigned mw = 0u - (unsigned)isW;
        const unsigned soff = (mw & ((32 * ss * NDIM + 64 * c) * 2)) | (~mw & (((xo * MLOC + 16 * u) * KLOC + peer * 64) * 2));
        const unsigned voff = (mw & ((lane >> 3) * (NDIM * 2) + (lane & 7) * 16)) | (~mw & ((lane & 15) * (KLOC * 2) + g4 * 16));
        const unsigned st1 = (mw & (8 * NDIM * 2)) | (~mw & 64), st2 = mw & (16 * NDIM * 2);
        const char* sbase = (isW ? wg : xa ? xg1 : xg0) + soff;
        u32x4 v[4];
        auto load = [&]() {
            v[0] = *(const GAS u32x4*)(sbase + voff);
            v[1] = *(const GAS u32x4*)(sbase + st1 + voff);
            v[2] = *(const GAS u32x4*)(sbase + st2 + voff);
            v[3] = *(const GAS u32x4*)(sbase + st1 + st2 + voff);
        };
        const bool fast = idx < NEARLY && wave < 4;
        if (fast) load();
        asm volatile("" ::: "memory");
        GAS unsigned* cnt = (GAS unsigned*)(me + CTRL_OFF + 4096) + (b * 8 + wave);
        const unsigned cval = __hip_atomic_load(cnt, __ATOMIC_RELAXED, __HIP_MEMORY_SCOPE_AGENT);
        unsigned tsend = isW ? p.tw[c] : p.tx[xslot];
        char* pws = p.crs[peer];
        unsigned slead = p.slead, sub16 = p.sub16, tz = p.tz, dz = p.dz;
        const bool sends = wave < 4 || (NSPLIT != 0 && wave < 6 && idx == 8);
        if (sends) {
            if (!fast) {
                const unsigned w0 = tsend & 0xffffu;
                if (w0 > slead) wait_until(t0, w0 - slead);
                load();
            }
            const unsigned when = (tsend & 0xffffu) + (((isW ? (ss & 1) * 2 : xo) * sub16) >> 4);
            if (isW) {
                char* sl = lds + wave * 4096;
#pragma unroll
                for (int q = 0; q < 4; ++q) *(u32x4*)(sl + q * 1024 + lane * 16) = v[q];
                const char* base = sl + (8 * g4 + q4) * 128 + pc * 16;
                v[0] = tr_frag(base, 512); v[1] = tr_frag(base + 8, 512);
                v[2] = tr_frag(base + 64, 512); v[3] = tr_frag(base + 72, 512);
            }
            const unsigned dst = (isW ? c * 8192u + (ss & 1) * 2048u : XIN + (xslot * 4 + ((xo - peer) & 3)) * 2048u) + lane * 16u;
#pragma unroll
            for (int q = 0; q < 4; ++q) v[q] = nz4(v[q]);
            auto rp = RSRC(pws + ((__builtin_amdgcn_readfirstlane(cval) + 1u) % 3u) * BUFB, ISRC);
            STAMP(1);
            wait_until(t0, when);
            __builtin_amdgcn_raw_buffer_store_b128(v[0], rp, dst, 0, AUX_SYS);
            __builtin_amdgcn_raw_buffer_store_b128(v[1], rp, dst + 1024, 0, AUX_SYS);
            if (isW) {
                __builtin_amdgcn_raw_buffer_store_b128(v[2], rp, dst + 4096, 0, AUX_SYS);
                __builtin_amdgcn_raw_buffer_store_b128(v[3], rp, dst + 4096 + 1024, 0, AUX_SYS);
            }
            STAMP(2);
        }
        unsigned cv2 = cval;
        asm volatile("" : "+v"(cv2));
        const unsigned call = __builtin_amdgcn_readfirstlane(cv2) + 1;
        if (lane == 0) __hip_atomic_store(cnt, call, __ATOMIC_RELAXED, __HIP_MEMORY_SCOPE_AGENT);
        if (wave >= 4) {
            auto rz = RSRC(me + ((call + 2) % 3) * BUFB, BUFB);
            const u32x4 z = {0u, 0u, 0u, 0u};
            const unsigned zo = ((idx * 8 + xcd) * 4 + (wave - 4)) * 1024u + lane * 16u;
            const unsigned zph = ((idx * 4 + (wave - 4)) * dz) >> 6;
#pragma unroll 1
            for (unsigned q = 0; q < (dz == 9999u ? 0u : 11u); ++q) {
                wait_until(t0, tz + zph + q * dz);
                __builtin_amdgcn_raw_buffer_store_b128(z, rz, zo + q * (512u << 10), 0, AUX_SYS);
            }
            if (idx == 0) __builtin_amdgcn_raw_buffer_store_b128(z, rz, zo + 11u * (512u << 10), 0, AUX_SYS);
        }
        DBG_FLUSH();
        return;
    }

    const unsigned u = (xcd >> 1) + 4 * (idx & 3);
    const unsigned jj = (idx >> 2) * 2 + (xcd & 1);
    const unsigned grp = wave >> 1, w2 = wave & 1;
    const unsigned c = jj + 8 * grp;
    const bool split = u == NA && jj < NSPLIT;
    const bool isA = u < NA || split;
    const unsigned xslot = split ? 16 : u;
    const unsigned ci = isA ? 0 : 1;
    const unsigned wlane = (lane >> 3) * (NDIM * 2) + ((lane & 7) ^ (((lane >> 4) & 1) << 2)) * 16;
    const char* xl0 = p.xl[ci][0]; const char* xl1 = p.xl[ci][1]; const char* xl2 = p.xl[ci][2]; const char* xl3 = p.xl[ci][3];
    unsigned r0 = p.cls[ci][0], rd = p.cls[ci][1], tpb = p.cls[ci][2], tps = p.cls[ci][3];
    {
        const unsigned zero = 0, fa = (unsigned)(size_t)(LAS char*)lds + L_FLAG + (lane & 15) * 4;
        asm volatile("ds_write_b32 %10, %11\n\ts_waitcnt lgkmcnt(0)\n\ts_barrier"
                     : "+s"(wg), "+s"(me), "+s"(xl0), "+s"(xl1), "+s"(xl2), "+s"(xl3), "+s"(r0), "+s"(rd), "+s"(tpb), "+s"(tps)
                     : "v"(fa), "v"(zero) : "memory");
    }
    const unsigned tp0 = tpb + jj * tps;
    if (grp == 1) {
        if (tp0) wait_until(t0, tp0);
        const unsigned xlane = (16 * u + (lane >> 3)) * (KLOC * 2) + ((lane & 7) ^ (lane >> 3)) * 16 + w2 * 128;
        const char* wsrc = wg + (64 * w2 * NDIM + 64 * jj) * 2;
#pragma unroll
        for (int h = 0; h < 2; ++h) {
#pragma unroll
            for (int i = 0; i < 8; ++i)
                GLDS(wsrc + (size_t)(h * 128 + i * 8) * NDIM * 2 + wlane, lds + L_W + h * 16384 + w2 * 8192 + i * 1024);
#pragma unroll
            for (int q = 0; q < 4; ++q) {
                const char* xs = (q == 0 ? xl0 : q == 1 ? xl1 : q == 2 ? xl2 : xl3) + h * 256;
                GLDS(xs + xlane, lds + L_XL + q * 8192 + h * 4096 + w2 * 2048);
                GLDS(xs + 8 * KLOC * 2 + xlane, lds + L_XL + q * 8192 + h * 4096 + w2 * 2048 + 1024);
            }
        }
    }
    GAS unsigned* cnt = (GAS unsigned*)(me + CTRL_OFF + 4096) + (b * 8 + wave);
    GAS unsigned* tmo = (GAS unsigned*)(me + CTRL_OFF);
    const unsigned cval = __hip_atomic_load(cnt, __ATOMIC_RELAXED, __HIP_MEMORY_SCOPE_AGENT);
    const unsigned tval = __hip_atomic_load(tmo, __ATOMIC_RELAXED, __HIP_MEMORY_SCOPE_AGENT);
    char* dst0 = p.dst[ci][0]; char* dst1 = p.dst[ci][1]; char* dst2 = p.dst[ci][2]; char* dst3 = p.dst[ci][3];
    char* outp = p.out;
    unsigned prefm = p.pref, rlead = p.rlead, rblk = p.rblk, rl2 = p.rl2, gmin = p.gmin, gmax = p.gmax, gend = p.gend, tb0 = p.tb0;
    unsigned sprint = p.sprint, sprc = p.sprc, dtx = p.dt[ci * 8 + xcd], tlc = p.tl[grp], tl0 = p.tl[0], tl1 = p.tl[1];
    unsigned txu = p.tx[xslot], twc = p.tw[c];
    asm volatile("" : "+s"(dst0), "+s"(dst1), "+s"(dst2), "+s"(dst3), "+s"(outp), "+s"(prefm), "+s"(rlead), "+s"(rblk), "+s"(rl2),
                      "+s"(gmin), "+s"(gmax), "+s"(gend), "+s"(tb0), "+s"(sprint), "+s"(sprc), "+s"(dtx), "+s"(tlc), "+s"(tl0), "+s"(tl1),
                      "+s"(txu), "+s"(twc));
    txu >>= 16;
    twc >>= 16;
    if (grp == 1) {
        asm volatile("s_waitcnt vmcnt(18)" ::: "memory");
        flag_add(lds, F_LA, lane);
    }
    const char* xl_rd0 = lds + L_XL + (lane & 15) * 128 + (g4 ^ (lane & 7)) * 16;
    const char* xl_rd1 = lds + L_XL + (lane & 15) * 128 + ((4 + g4) ^ (lane & 7)) * 16;
    const char* w_rd = lds + L_W + (grp & 1) * 32768 + (8 * g4 + q4) * 128 + ((w2 ^ (q4 >> 1)) * 64) + pc * 16;
    const unsigned wroff = IOFF + (c * 2 + w2) * 4096u + lane * 16u;
    const unsigned toff = POFF + ((((c >> 1) * 16 + u) * 2 + (c & 1)) * 2 + w2) * 1024u + lane * 16u;
    u32x4 wr[4][4];
#pragma unroll
    for (int k = 0; k < 16; ++k) wr[k >> 2][k & 3] = u32x4{0u, 0u, 0u, 0u};
    WAIT_VM0();
    if (grp == 1) flag_add(lds, F_LB, lane);
    const unsigned call = __builtin_amdgcn_readfirstlane(cval) + 1;
    const unsigned bufoff = (call % 3) * BUFB;
    bool bad = __builtin_amdgcn_readfirstlane(tval) != 0;
    if (lane == 0) __hip_atomic_store(cnt, call, __ATOMIC_RELAXED, __HIP_MEMORY_SCOPE_AGENT);
    auto rl = RSRC(me + bufoff, BUFB);
    auto give_up = [&]() -> bool {
        const unsigned el = now32() - t0;
        if (!bad && el < 10000u) return false;
        if (bad || el > DEADLINE_TICKS ||
            __builtin_amdgcn_readfirstlane(__hip_atomic_load(tmo, __ATOMIC_RELAXED, __HIP_MEMORY_SCOPE_AGENT)) != 0) {
            if (lane == 0) __hip_atomic_store(tmo, call, __ATOMIC_RELAXED, __HIP_MEMORY_SCOPE_AGENT);
            bad = true;
            return true;
        }
        __builtin_amdgcn_s_sleep(64);
        return false;
    };
    if (grp >= 2) {
        const unsigned src = wave - 4;
        auto load_w1 = [&]() {
            wait_until(t0, tp0 + tl1);
            const char* wsrc = wg + (64 * src * NDIM + 64 * (jj + 8)) * 2;
#pragma unroll
            for (int i = 0; i < 8; ++i) GLDS(wsrc + (size_t)i * 8 * NDIM * 2 + wlane, lds + L_W + 32768 + src * 8192 + i * 1024);
            WAIT_VM0();
            flag_add(lds, F_WL + 1, lane);
        };
        const bool xfirst = txu <= tp0 + tl0;
        if (!xfirst) load_w1();
        wait_until(t0, txu);
        u32x4 xr[8];
        for (;;) {
            unsigned mn = 0xffffffffu;
#pragma unroll
            for (int i = 0; i < 8; ++i)
                xr[i] = __builtin_amdgcn_raw_buffer_load_b128(rl, IOFF + src * ISRC + XIN + xslot * 8192u + i * 1024u + lane * 16u, 0, AUX_SYS);
#pragma unroll
            for (int i = 0; i < 8; ++i) mn = min(mn, min4(xr[i]));
            if (__all(mn != 0)) break;
            if (give_up()) {
#pragma unroll
                for (int i = 0; i < 8; ++i) xr[i] = u32x4{NAN2, NAN2, NAN2, NAN2};
                break;
            }
            __builtin_amdgcn_s_sleep(XSL);
        }
#pragma unroll
        for (int i = 0; i < 8; ++i) *(u32x4*)(lds + L_XR + src * 8192 + i * 1024 + lane * 16) = xr[i];
        WAIT_LGKM0();
        flag_add(lds, F_XR + src, lane);
        STAMP(3);
        if (xfirst) load_w1();
        flag_wait(lds, F_FREE + (grp & 1), 2);
        wait_until(t0, tlc);
        const char* wsrc = wg + (128 * w2 * NDIM + 64 * c) * 2;
#pragma unroll
        for (int i = 0; i < 16; ++i) GLDS(wsrc + (size_t)i * 8 * NDIM * 2 + wlane, lds + L_W + (grp & 1) * 32768 + w2 * 16384 + i * 1024);
        WAIT_VM0();
        flag_add(lds, F_WL + grp, lane);
    }
    const u32x4 rlw = {(unsigned)(size_t)(me + bufoff), (unsigned)((size_t)(me + bufoff) >> 32) & 0xffffu, BUFB, 0x00020000u};
#define WR_ALL "+v"(wr[0][0]), "+v"(wr[0][1]), "+v"(wr[0][2]), "+v"(wr[0][3]), "+v"(wr[1][0]), "+v"(wr[1][1]), "+v"(wr[1][2]), "+v"(wr[1][3]), \
               "+v"(wr[2][0]), "+v"(wr[2][1]), "+v"(wr[2][2]), "+v"(wr[2][3]), "+v"(wr[3][0]), "+v"(wr[3][1]), "+v"(wr[3][2]), "+v"(wr[3][3])
    auto load_wr = [&](unsigned mask) {
        const unsigned s1 = ISRC, s2 = 2 * ISRC, s3 = 3 * ISRC;
#define LD4(a, b_, c_, d_, so) \
        "buffer_load_dwordx4 %" #a ", %16, %17, " so " offen sc0 sc1\n\t" \
        "buffer_load_dwordx4 %" #b_ ", %16, %17, " so " offen offset:1024 sc0 sc1\n\t" \
        "buffer_load_dwordx4 %" #c_ ", %16, %17, " so " offen offset:2048 sc0 sc1\n\t" \
        "buffer_load_dwordx4 %" #d_ ", %16, %17, " so " offen offset:3072 sc0 sc1\n\t"
        asm volatile("s_nop 4\n\t"
                     "s_bitcmp0_b32 %18, 0\n\ts_cbranch_scc1 1f\n\t" LD4(0, 1, 2, 3, "0")
                     "1:\n\ts_bitcmp0_b32 %18, 1\n\ts_cbranch_scc1 2f\n\t" LD4(4, 5, 6, 7, "%19")
                     "2:\n\ts_bitcmp0_b32 %18, 2\n\ts_cbranch_scc1 3f\n\t" LD4(8, 9, 10, 11, "%20")
                     "3:\n\ts_bitcmp0_b32 %18, 3\n\ts_cbranch_scc1 4f\n\t" LD4(12, 13, 14, 15, "%21")
                     "4:"
                     : WR_ALL : "v"(wroff), "s"(rlw), "s"(mask), "s"(s1), "s"(s2), "s"(s3) : "memory", "scc");
    };
    auto wait_wr = [&]() { asm volatile("s_waitcnt vmcnt(0)" : WR_ALL :: "memory"); };
    auto wmin = [&](int sc) { return min(min(min4(wr[sc][0]), min4(wr[sc][1])), min(min4(wr[sc][2]), min4(wr[sc][3]))); };
    const bool pref = twc <= max(prefm, now32() - t0 + 25u);
    if (pref) wait_until(t0, twc);
    load_wr(__builtin_amdgcn_readfirstlane(pref ? 15u : 0u));
    unsigned tres = r0 + (((c & ~1u) * (rd >> 16) + 2u * (isA ? u : u - NA) * (rd & 0xffffu)) >> 4);
    unsigned xr_a = (unsigned)(size_t)(LAS char*)lds + L_XR + lane * 16;
    unsigned fa = (unsigned)(size_t)(LAS char*)lds + L_FLAG + F_XR * 4 + (lane & 3) * 4;
    char* pacer0 = me + CTRL_OFF + 1024 + (ci * 8 + (xcd & sprint)) * 128;
    unsigned long long tmin = isA ? 0ull : t064 + tb0;
    asm volatile("" : "+s"(tres), "+s"(dtx), "+s"(pacer0), "+s"(tmin), "+v"(xr_a), "+v"(fa));
    if (grp >= 1) { flag_wait(lds, F_WL + grp, grp == 1 ? 4 : 2); flag_wait(lds, F_LB, 2); }
    flag_wait(lds, F_LA, 2);
    STAMP(1);

    f32x4 acc[4][2];
    unsigned wm[4];
    {
#pragma unroll
        for (int q = 0; q < 4; ++q) { acc[q][0] = f32x4{0.f, 0.f, 0.f, 0.f}; acc[q][1] = acc[q][0]; }
        u32x4 wf[2][2], xf[2][4];
        auto rdl = [&](int s) {
            wf[s & 1][0] = tr_frag(w_rd + s * 4096, 512);
            xf[s & 1][0] = *(const u32x4*)(((s & 1) ? xl_rd1 : xl_rd0) + (0 * 4 + (s >> 1)) * 2048);
            wf[s & 1][1] = tr_frag(w_rd + s * 4096 + 8, 512);
#pragma unroll
            for (int q = 1; q < 4; ++q)
                xf[s & 1][q] = *(const u32x4*)(((s & 1) ? xl_rd1 : xl_rd0) + (q * 4 + (s >> 1)) * 2048);
        };
        rdl(0);
        SB();
#pragma unroll
        for (int s = 0; s < 8; ++s) {
            if (s == 3) { flag_wait(lds, F_LB, 2); SB(); }
            if (s == 6) { wait_wr(); SB(); }
            if (s + 1 < 8) rdl(s + 1);
#pragma unroll
            for (int q = 0; q < 4; ++q) {
                acc[q][0] = MFMA(wf[s & 1][0], xf[s & 1][q], acc[q][0]);
                acc[q][1] = MFMA(wf[s & 1][1], xf[s & 1][q], acc[q][1]);
            }
            if (s >= 6) { wm[2 * (s - 6)] = wmin(2 * (s - 6)); wm[2 * (s - 6) + 1] = wmin(2 * (s - 6) + 1); }
            if (s < 6) {
                SGB(0x100, 2); SGB(0x8, 1); SGB(0x100, 2); SGB(0x8, 1); SGB(0x100, 1); SGB(0x8, 1); SGB(0x100, 1); SGB(0x8, 1);
                SGB(0x100, 1); SGB(0x8, 1); SGB(0x100, 1); SGB(0x8, 3);
            } else if (s == 6) {
                SGB(0x100, 2); SGB(0x8, 1); SGB(0x2, 2); SGB(0x100, 2); SGB(0x8, 1); SGB(0x2, 2); SGB(0x100, 1); SGB(0x8, 1); SGB(0x2, 2);
                SGB(0x100, 1); SGB(0x8, 1); SGB(0x2, 2); SGB(0x100, 1); SGB(0x8, 1); SGB(0x2, 2); SGB(0x100, 1); SGB(0x8, 1); SGB(0x2, 2);
                SGB(0x8, 1); SGB(0x2, 2); SGB(0x8, 1); SGB(0x2, 2);
            } else {
                SGB(0x8, 1); SGB(0x2, 2); SGB(0x8, 1); SGB(0x2, 2); SGB(0x8, 1); SGB(0x2, 2); SGB(0x8, 1); SGB(0x2, 2);
                SGB(0x8, 1); SGB(0x2, 2); SGB(0x8, 1); SGB(0x2, 2); SGB(0x8, 1); SGB(0x2, 2); SGB(0x8, 1); SGB(0x2, 2);
            }
            SB();
            if (s >= 6) asm volatile("" : "+v"(wm[2 * (s - 6)]), "+v"(wm[2 * (s - 6) + 1]));
        }
        if (grp < 2) flag_add(lds, F_FREE + grp, lane);
    }
    STAMP(2);
    unsigned long long slot = 0;
    auto reserve = [&](unsigned long long want) {
        unsigned vz = 0;
        asm volatile("" : "+v"(vz));
        GAS unsigned long long* pacer = (GAS unsigned long long*)(pacer0 + vz);
        if (want < tmin) want = tmin;
        if (lane == 0) {
            __hip_atomic_fetch_max(pacer, want << 8, __ATOMIC_RELAXED, __HIP_MEMORY_SCOPE_WORKGROUP);
            slot = __hip_atomic_fetch_add(pacer, (unsigned long long)dtx, __ATOMIC_RELAXED, __HIP_MEMORY_SCOPE_WORKGROUP);
        }
    };
    bool early = false;
    {
        unsigned done = 0;
        for (;;) {
            unsigned fl;
            asm volatile("ds_read_b32 %0, %1" : "=v"(fl) : "v"(fa) : "memory");
            unsigned wok = 0;
#pragma unroll
            for (int sc = 0; sc < 4; ++sc) if (__all(wm[sc] != 0)) wok |= 1u << sc;
            wok = __builtin_amdgcn_readfirstlane(wok);
            if (wok == 0) wait_until(t0, twc);
            load_wr(~wok & 15u);
            asm volatile("s_waitcnt lgkmcnt(0)" : "+v"(fl) :: "memory");
            const unsigned ready = wok & (unsigned)__builtin_amdgcn_ballot_w64(fl != 0) & ~done & 15u;
            if (ready) {
                u32x4 xf[8];
                const unsigned a0 = xr_a + __builtin_ctz(ready) * 8192;
#pragma unroll
                for (int k = 0; k < 8; ++k) xf[k] = *(const LAS u32x4*)(size_t)(a0 + k * 1024);
                SB();
                if ((ready | done) == 15u) {
                    STAMP(5);
                    const unsigned long long tn = __builtin_amdgcn_s_memrealtime();
                    const unsigned ahead = rlead + rblk * __builtin_popcount(ready);
                    early = (unsigned)tn - t0 + ahead >= tres;
                    if (early) reserve(tn + ahead);
                }
#pragma unroll
                for (int sc = 0; sc < 4; ++sc)
                    if ((ready >> sc) & 1) {
                        const unsigned an = xr_a + (__builtin_ctz((ready >> (sc + 1)) | 8u) + sc + 1 & 3) * 8192;
#pragma unroll
                        for (int k = 0; k < 8; ++k) {
                            acc[k >> 1][0] = MFMA(wr[sc][(k & 1) * 2], xf[k], acc[k >> 1][0]);
                            acc[k >> 1][1] = MFMA(wr[sc][(k & 1) * 2 + 1], xf[k], acc[k >> 1][1]);
                            if (sc < 3) xf[k] = *(const LAS u32x4*)(size_t)(an + k * 1024);
                            SB();
                        }
                    }
            }
            done |= ready;
            if (done == 15u) break;
            const bool gu = give_up();
            wait_wr();
            if (gu) break;
            if (wok == 15u) __builtin_amdgcn_s_sleep(1);
            else {
#pragma unroll
                for (int sc = 0; sc < 4; ++sc) wm[sc] = wmin(sc);
            }
        }
    }
    unsigned gate;
    {
        u32x4 d[4];
#pragma unroll
        for (int q = 0; q < 4; ++q)
#pragma unroll
            for (int k = 0; k < 4; ++k) d[q][k] = nz(pack2(acc[q][k >> 1][(2 * k) & 3], acc[q][k >> 1][(2 * k + 1) & 3]));
        if (bad) {
#pragma unroll
            for (int q = 0; q < 4; ++q) d[q] = u32x4{NAN2, NAN2, NAN2, NAN2};
        }
        auto push = [&](int q) {
            auto rs = RSRC((q == 0 ? dst0 : q == 1 ? dst1 : q == 2 ? dst2 : dst3) + bufoff, 1u << 20);
            __builtin_amdgcn_raw_buffer_store_b128(d[q], rs, toff, 0, AUX_SYS);
        };
        STAMP(6);
        if (!early) {
            wait_until(t0, tres);
            bool again = true;
            if (c >= sprc) {
                unsigned vz = 0;
                asm volatile("" : "+v"(vz));
                if (lane == 0) slot = __hip_atomic_fetch_add((GAS unsigned long long*)(pacer0 + vz), (unsigned long long)dtx, __ATOMIC_RELAXED, __HIP_MEMORY_SCOPE_WORKGROUP);
                again = (unsigned)__builtin_amdgcn_readfirstlane((unsigned)(slot >> 8)) - now32() >= 4000u;
            }
            if (again) reserve(__builtin_amdgcn_s_memrealtime() + rl2);
        }
        asm volatile("" : "+v"(slot));
        gate = __builtin_amdgcn_readfirstlane((unsigned)(slot >> 8)) - t0;
        for (;;) {
            const unsigned tn = now32() - t0;
            if (gate - tn - 1u >= 4000u) { gate = tn; break; }
            __builtin_amdgcn_s_sleep(1);
        }
        push(1); push(2); push(3);
        if (!isA) push(0);
    }
    STAMP(7);
    const unsigned so0 = isA ? 1u << 20 : 0u;
    GAS u32x4* optr = (GAS u32x4*)(outp + ((u * 16 + (lane & 15)) * NDIM + c * 64 + w2 * 32 + g4 * 8) * 2);
    u32x4 in[4];
    wait_until(t0, max(gate + gmin, min(gate + gmax, gend)));
    for (;;) {
        in[0] = __builtin_amdgcn_raw_buffer_load_b128(rl, toff, so0, AUX_SYS);
        in[1] = __builtin_amdgcn_raw_buffer_load_b128(rl, toff, 1u << 20, AUX_SYS);
        in[2] = __builtin_amdgcn_raw_buffer_load_b128(rl, toff, 2u << 20, AUX_SYS);
        in[3] = __builtin_amdgcn_raw_buffer_load_b128(rl, toff, 3u << 20, AUX_SYS);
        if (__all(min(min(min4(in[0]), min4(in[1])), min(min4(in[2]), min4(in[3]))) != 0)) break;
        if (give_up()) break;
    }
    STAMP(9);
    float y[8];
#pragma unroll
    for (int k = 0; k < 4; ++k) {
        y[2 * k] = isA ? acc[0][k >> 1][(2 * k) & 3] : lo_f(in[0][k]);
        y[2 * k + 1] = isA ? acc[0][k >> 1][(2 * k + 1) & 3] : hi_f(in[0][k]);
    }
#pragma unroll
    for (int q = 1; q < 4; ++q)
#pragma unroll
        for (int k = 0; k < 4; ++k) {
            y[2 * k] += lo_f(in[q][k]);
            y[2 * k + 1] += hi_f(in[q][k]);
        }
    u32x4 o = {pack2(relu(y[0]), relu(y[1])), pack2(relu(y[2]), relu(y[3])),
               pack2(relu(y[4]), relu(y[5])), pack2(relu(y[6]), relu(y[7]))};
    if (bad) o = u32x4{NAN2, NAN2, NAN2, NAN2};
    *optr = o;
}

extern "C" void kernel_launch(int rank, int nranks, ncclComm_t comm,
                              hipStream_t stream,
                              void* const* d_in, const int* in_sizes, int n_in,
                              void* d_out, int out_size,
                              const int* in_dtypes, int out_dtype,
                              void* const* d_out_all,
                              void* const* d_ws_all, size_t ws_bytes) {
    (void)comm; (void)in_dtypes; (void)out_dtype; (void)d_out_all;
    if (nranks != NR || (unsigned)rank >= (unsigned)NR || n_in < 2 || ws_bytes < WS_NEED ||
        in_sizes[0] != MDIM * KLOC || in_sizes[1] != KLOC * NDIM || out_size != MLOC * NDIM) {
        fprintf(stderr, "kernel_launch: unexpected problem shape\n");
        return;
    }
    Params p = {};
    const int g4 = rank & 4, gi = rank & 3;
    const char* x = (const char*)d_in[0];
    const size_t own = (size_t)MLOC * KLOC * 2;
    p.w = (const char*)d_in[1];
    p.out = (char*)d_out;
    p.me = (char*)d_ws_all[rank];
    p.xg[0] = x + g4 * own;
    p.xg[1] = x + (g4 ^ 4) * own;
    for (int q = 0; q < 4; ++q) {
        const int o = (gi + q) & 3;
        p.xl[0][q] = x + (g4 + o) * own;
        p.xl[1][q] = x + ((g4 ^ 4) + o) * own;
        p.dst[0][q] = (char*)d_ws_all[g4 + o] + POFF + ((size_t)((4 - q) & 3) << 20);
        p.dst[1][q] = (char*)d_ws_all[(g4 ^ 4) + o] + POFF + ((size_t)((4 - q) & 3) << 20);
        p.crs[q] = (char*)d_ws_all[(g4 ^ 4) + q] + IOFF + (size_t)gi * ISRC;
    }
    unsigned t_first = 80, unit16 = 220;
    const char* order = "XXWWXXWWXWXWXWXWXWXWWWXWXWXWXWXWXWXWWWWWWWWWWWWWWW";
    p.tz = 200; p.dz = 110; p.slead = 130; p.pref = 0;
    p.rlead = 15; p.rblk = 14; p.rl2 = 30; p.gmin = 50; p.gmax = 250; p.gend = 1450; p.tb0 = 760; p.sprint = 7; p.sprc = 99;
    p.tl[0] = 200; p.tl[1] = 95; p.tl[2] = 400; p.tl[3] = 460;
    unsigned da = 56, db = 56, lead = 40, leadmax = 110, tlb = 150, tlj = 10, nfast = 10, ra0 = 200, rb0 = 660;
    p.sub16 = unit16 / 4;
    const unsigned nwa = 8 * NA + NSPLIT, nwb = 128 - nwa;
    p.cls[0][0] = ra0; p.cls[0][1] = da | ((da * nwa / 8) << 16); p.cls[0][2] = 0; p.cls[0][3] = tlj;
    p.cls[1][0] = rb0; p.cls[1][1] = db | ((db * nwb / 8) << 16); p.cls[1][2] = tlb; p.cls[1][3] = 0;
    const int nxs = NSPLIT ? 17 : 16;
    int nx = 0, nw = 0; unsigned pos = 0;
    for (const char* s = order; *s; ++s) {
        if (*s == 'X' && nx < nxs) {
            const unsigned t = t_first + ((pos++ * unit16) >> 4);
            const int slot = !NSPLIT ? nx : nx < NA ? nx : nx == NA ? 16 : nx - 1;
            p.tx[slot] = t | ((t + ((slot < NA || slot == 16) ? lead : leadmax)) << 16); ++nx;
        } else if (*s == 'W' && nw < 32) {
            const unsigned t = t_first + ((pos++ * unit16) >> 4);
            p.tw[nw] = t | ((t + (nw < (int)nfast ? lead : leadmax)) << 16); ++nw;
        }
    }
    if (nx != nxs || nw != 32) { fprintf(stderr, "bad order\n"); return; }
    for (int xc = 0; xc < 8; ++xc) {
        unsigned ra = 0, rb = 0;
        for (int idx = 0; idx < 16; ++idx) {
            for (int x2 = 0; x2 < 8; ++x2) {
                if ((x2 & p.sprint) != (xc & p.sprint)) continue;
                const int u = (x2 >> 1) + 4 * (idx & 3), jj = (idx >> 2) * 2 + (x2 & 1);
                ((u < NA || (u == NA && jj < NSPLIT)) ? ra : rb)++;
            }
        }
        p.dt[xc] = ra ? da * nwa * 8 / ra : 0;
        p.dt[8 + xc] = rb ? db * nwb * 8 / rb : 0;
    }
    gemm_rs_kernel<<<256, 512, 0, stream>>>(p);
    HIP_CHECK(hipGetLastError());
}
```

```cpp
#include <hip/hip_runtime.h>
#include <rccl/rccl.h>
#include <cstdio>
#include <cstdint>
#include <cstdlib>

#define HIP_CHECK(expr) do { hipError_t _e = (expr); if (_e != hipSuccess) { \
    fprintf(stderr, "HIP error %s at %s:%d\n", hipGetErrorString(_e), __FILE__, __LINE__); \
    return; } } while (0)

typedef __attribute__((ext_vector_type(8))) __bf16 bf16x8;
typedef __attribute__((ext_vector_type(4))) __bf16 bf16x4;
typedef __attribute__((ext_vector_type(2))) __bf16 bf16x2;
typedef __attribute__((ext_vector_type(2))) float f32x2;
typedef __attribute__((ext_vector_type(4))) float f32x4;
typedef __attribute__((ext_vector_type(4))) unsigned u32x4;
typedef __attribute__((ext_vector_type(2))) unsigned u32x2;
#define LAS __attribute__((address_space(3)))
#define GAS __attribute__((address_space(1)))

#define NA 9
#define NSPLIT 4
#define XSL 0
#define KF 4u
#define T_BIAS 100u
#define SCOUT 5
#define NEARLY 4

constexpr int NR = 8;
constexpr int MDIM = 2048, NDIM = 2048, KLOC = 256, MLOC = 256;
constexpr size_t CTRL_OFF = 0;
constexpr unsigned RA_ = NA + 1, RB_ = 16 - NA;
constexpr unsigned XIN = 32u * 8192u;
constexpr unsigned PBO = XIN + 17u * 8192u;
constexpr unsigned CBUF = PBO + RB_ * 65536u;
constexpr unsigned ABUF = RA_ * 65536u;
constexpr unsigned COFF = 64u << 10;
constexpr unsigned AOFF = COFF + 12u * CBUF - 3u * ABUF;
constexpr unsigned WS_DATA = AOFF + 12u * ABUF;
constexpr size_t WS_NEED = WS_DATA;
constexpr unsigned DEADLINE_TICKS = 100000000u;
constexpr int AUX_SYS = 17;
constexpr int L_W = 0, L_XL = 65536, L_XR = 98304, L_FLAG = 131072, L_TOTAL = 131072 + 64;
constexpr unsigned NAN2 = 0x7fc07fc0u;

struct Params {
    const char* w;
    char* me;
    const char* xg[2];
    const char* xl[2][4];
    char* dst[2][4];
    char* crs[4];
    char* out;
    unsigned tz, dz;
    unsigned slead, sub16, pref, rlead, rblk, rl2, gmin, gmax;
    unsigned gend, tb0, sprint, sprc;
    unsigned cls[2][4];
    unsigned dt[16];
    unsigned tx[20];
    unsigned tw[32];
    unsigned tl[4];
    unsigned ttch, pad;
    char* tch[12];
};

__device__ __forceinline__ unsigned pack2(float a, float b) {
    f32x2 v = {a, b};
    return __builtin_bit_cast(unsigned, __builtin_convertvector(v, bf16x2));
}
__device__ __forceinline__ unsigned nz(unsigned d) { return d | ((d - 1u) & 0x80000000u); }
__device__ __forceinline__ u32x4 nz4(u32x4 d) { return u32x4{nz(d[0]), nz(d[1]), nz(d[2]), nz(d[3])}; }
__device__ __forceinline__ float lo_f(unsigned d) { return __builtin_bit_cast(float, d << 16); }
__device__ __forceinline__ float hi_f(unsigned d) { return __builtin_bit_cast(float, d & 0xffff0000u); }
__device__ __forceinline__ float relu(float y) { return y < 0.f ? 0.f : y; }
__device__ __forceinline__ unsigned min4(u32x4 v) { return min(min(v[0], v[1]), min(v[2], v[3])); }
__device__ __forceinline__ unsigned now32() { return (unsigned)__builtin_amdgcn_s_memrealtime(); }
__device__ __forceinline__ void wait_until(unsigned t0, unsigned d) {
    while (now32() - t0 < d) __builtin_amdgcn_s_sleep(1);
}
__device__ __forceinline__ u32x4 tr_frag(const char* a, int hi_off) {
    u32x2 lo = __builtin_bit_cast(u32x2, __builtin_amdgcn_ds_read_tr16_b64_v4bf16((LAS bf16x4*)a));
    u32x2 hi = __builtin_bit_cast(u32x2, __builtin_amdgcn_ds_read_tr16_b64_v4bf16((LAS bf16x4*)(a + hi_off)));
    return u32x4{lo[0], lo[1], hi[0], hi[1]};
}
#define MFMA(a, bb, c) __builtin_amdgcn_mfma_f32_16x16x32_bf16(__builtin_bit_cast(bf16x8, a), __builtin_bit_cast(bf16x8, bb), c, 0, 0, 0)
#define WAIT_VM0() __builtin_amdgcn_s_waitcnt(0x0F70)
#define WAIT_LGKM0() __builtin_amdgcn_s_waitcnt(0xC07F)
#define SGB(mask, n) __builtin_amdgcn_sched_group_barrier(mask, n, 0)
#define SB() __builtin_amdgcn_sched_barrier(0)
#define RSRC(base, bytes) __builtin_amdgcn_make_buffer_rsrc((base), 0, (int)(bytes), 0x00020000)
#define GLDS(src, dstlds) __builtin_amdgcn_global_load_lds((const GAS void*)(src), (LAS void*)(dstlds), 16, 0, 0)

__device__ __forceinline__ void flag_add(char* lds, int f, int lane) {
    const unsigned a = (unsigned)(size_t)(LAS char*)lds + L_FLAG + f * 4, one = 1;
    if (lane == 0) asm volatile("ds_add_u32 %0, %1" :: "v"(a), "v"(one) : "memory");
}
__device__ __forceinline__ void flag_wait(char* lds, int f, unsigned n) {
    const unsigned a = (unsigned)(size_t)(LAS char*)lds + L_FLAG + f * 4;
    for (;;) {
        unsigned v;
        asm volatile("ds_read_b32 %0, %1\n\ts_waitcnt lgkmcnt(0)" : "=&v"(v) : "v"(a) : "memory");
        if ((unsigned)__builtin_amdgcn_readfirstlane(v) >= n) break;
        __builtin_amdgcn_s_sleep(1);
    }
}
enum { F_XR = 0  , F_LA = 4, F_LB = 5, F_WL = 6  , F_FREE = 10   };

#define STAMP(i) do {} while (0)
#define DBG_FLUSH() do {} while (0)

__global__ __launch_bounds__(512, 2) void gemm_rs_kernel(Params p) {
    __shared__ __attribute__((aligned(16))) char lds[L_TOTAL];
    unsigned long long ta, tb;
    asm volatile("s_memrealtime %0" : "=s"(ta));
    SB();
    const char* wg = p.w; char* me = p.me; const char* xg0 = p.xg[0]; const char* xg1 = p.xg[1];
#define TIME_BASE() \
    const unsigned long long t064 = tb + min(((unsigned)(tb - ta) * KF) >> 3, T_BIAS) - T_BIAS; \
    const unsigned t0 = (unsigned)t064
    const unsigned lane = threadIdx.x & 63;
    const unsigned wave = __builtin_amdgcn_readfirstlane(threadIdx.x >> 6);
    const unsigned b = blockIdx.x;
    const unsigned xcd = b & 7, idx = b >> 4;
    const unsigned g4 = lane >> 4, q4 = (lane >> 2) & 3, pc = lane & 3;

    if (__builtin_expect((b & 8) == 0, 1)) {
        const bool isW = wave < 2;
        const unsigned c = 2 * idx + (wave & 1), ss = xcd;
        const unsigned peer = xcd >> 1, xo = (xcd & 1) * 2 + (wave & 1);
        const unsigned xslot = wave < 4 ? idx : 16, u = wave < 4 ? idx : NA;
        const bool xa = idx < NA || wave >= 4;
        const unsigned mw = 0u - (unsigned)isW;
        const unsigned soff = (mw & ((32 * ss * NDIM + 64 * c) * 2)) | (~mw & (((xo * MLOC + 16 * u) * KLOC + peer * 64) * 2));
        const unsigned voff = (mw & ((lane >> 3) * (NDIM * 2) + (lane & 7) * 16)) | (~mw & ((lane & 15) * (KLOC * 2) + g4 * 16));
        const unsigned st1 = (mw & (8 * NDIM * 2)) | (~mw & 64), st2 = mw & (16 * NDIM * 2);
        const char* sbase = (isW ? wg : xa ? xg1 : xg0) + soff;
        asm volatile("s_memrealtime %0" : "=s"(tb), "+s"(sbase));
        u32x4 v[4];
        auto load = [&]() {
            v[0] = *(const GAS u32x4*)(sbase + voff);
            v[1] = *(const GAS u32x4*)(sbase + st1 + voff);
            v[2] = *(const GAS u32x4*)(sbase + st2 + voff);
            v[3] = *(const GAS u32x4*)(sbase + st1 + st2 + voff);
        };
        const bool fast = idx < NEARLY && wave < 4;
        if (fast) load();
        asm volatile("s_waitcnt lgkmcnt(0)" : "+s"(ta), "+s"(tb) :: "memory");
        TIME_BASE();
        char* pws = p.crs[peer];
        if (wave == 4 && idx == SCOUT) {
            unsigned sink = __hip_atomic_load((const GAS unsigned*)me, __ATOMIC_RELAXED, __HIP_MEMORY_SCOPE_AGENT);
            sink += __hip_atomic_load((const GAS unsigned*)(pws + (lane & 1) * (3 * CBUF - 64)), __ATOMIC_RELAXED, __HIP_MEMORY_SCOPE_SYSTEM);
            asm volatile("" :: "v"(sink));
            const unsigned ttch = p.ttch;
            if (ttch != 9999u) wait_until(t0, ttch);
#pragma unroll 1
            for (int i = 0; i < (ttch != 9999u ? 12 : 0); ++i) {
                const unsigned len = i < 3 ? 0u : i < 6 ? 3 * ABUF - 64 : i < 10 ? 3 * CBUF - 64 : 0u;
                const unsigned t_ = __hip_atomic_load((const GAS unsigned*)(p.tch[i] + (lane & 1) * len), __ATOMIC_RELAXED, __HIP_MEMORY_SCOPE_SYSTEM);
                asm volatile("" :: "v"(t_));
            }
        }
        GAS unsigned* cnt = (GAS unsigned*)(me + CTRL_OFF + 4096) + (b * 8 + wave);
        const unsigned cval = __hip_atomic_load(cnt, __ATOMIC_RELAXED, __HIP_MEMORY_SCOPE_AGENT);
        unsigned tsend = isW ? p.tw[c] : p.tx[xslot];
        unsigned slead = p.slead, sub16 = p.sub16, tz = p.tz, dz = p.dz;
        const bool sends = wave < 4 || (NSPLIT != 0 && wave < 6 && idx == 8);
        if (sends) {
            if (!fast) {
                const unsigned w0 = tsend & 0xffffu;
                if (w0 > slead) wait_until(t0, w0 - slead);
                load();
            }
            const unsigned when = (tsend & 0xffffu) + (((isW ? (ss & 1) * 2 : xo) * sub16) >> 4);
            if (isW) {
                char* sl = lds + wave * 4096;
#pragma unroll
                for (int q = 0; q < 4; ++q) *(u32x4*)(sl + q * 1024 + lane * 16) = v[q];
                const char* base = sl + (8 * g4 + q4) * 128 + pc * 16;
                v[0] = tr_frag(base, 512); v[1] = tr_frag(base + 8, 512);
                v[2] = tr_frag(base + 64, 512); v[3] = tr_frag(base + 72, 512);
            }
            const unsigned dst = (isW ? c * 8192u + (ss & 1) * 2048u : XIN + (xslot * 4 + ((xo - peer) & 3)) * 2048u) + lane * 16u;
#pragma unroll
            for (int q = 0; q < 4; ++q) v[q] = nz4(v[q]);
            auto rp = RSRC(pws + ((__builtin_amdgcn_readfirstlane(cval) + 1u) % 3u) * CBUF, PBO);
            STAMP(1);
            wait_until(t0, when);
            __builtin_amdgcn_raw_buffer_store_b128(v[0], rp, dst, 0, AUX_SYS);
            __builtin_amdgcn_raw_buffer_store_b128(v[1], rp, dst + 1024, 0, AUX_SYS);
            if (isW) {
                __builtin_amdgcn_raw_buffer_store_b128(v[2], rp, dst + 4096, 0, AUX_SYS);
                __builtin_amdgcn_raw_buffer_store_b128(v[3], rp, dst + 4096 + 1024, 0, AUX_SYS);
            }
            STAMP(2);
        }
        unsigned cv2 = cval;
        asm volatile("" : "+v"(cv2));
        const unsigned call = __builtin_amdgcn_readfirstlane(cv2) + 1;
        if (lane == 0) __hip_atomic_store(cnt, call, __ATOMIC_RELAXED, __HIP_MEMORY_SCOPE_AGENT);
        if (wave >= 4) {
            auto rz = RSRC(me, WS_DATA);
            const u32x4 z = {0u, 0u, 0u, 0u};
            const unsigned zb = (call + 2) % 3, zid = (idx * 8 + xcd) * 4 + (wave - 4);
            const unsigned zph = ((idx * 4 + (wave - 4)) * dz) >> 6;
            constexpr unsigned NCP = CBUF / 1024, NAP = ABUF / 1024;
#pragma unroll 1
            for (unsigned q = 0; q < (dz == 9999u ? 0u : 11u); ++q) {
                const unsigned id = q * 512 + zid;
                if (id >= 4 * NCP + 3 * NAP) break;
                const unsigned i2 = id - 4 * NCP;
                const unsigned off = id < 4 * NCP ? COFF + ((id / NCP) * 3 + zb) * CBUF + (id % NCP) * 1024u
                                                  : AOFF + ((i2 / NAP + 1) * 3 + zb) * ABUF + (i2 % NAP) * 1024u;
                wait_until(t0, tz + zph + q * dz);
                __builtin_amdgcn_raw_buffer_store_b128(z, rz, off + lane * 16u, 0, AUX_SYS);
            }
        }
        DBG_FLUSH();
        return;
    }

    const unsigned u = (xcd >> 1) + 4 * (idx & 3);
    const unsigned jj = (idx >> 2) * 2 + (xcd & 1);
    const unsigned grp = wave >> 1, w2 = wave & 1;
    const unsigned c = jj + 8 * grp;
    const bool split = u == NA && jj < NSPLIT;
    const bool isA = u < NA || split;
    const unsigned xslot = split ? 16 : u;
    const unsigned ci = isA ? 0 : 1;
    const unsigned wlane = (lane >> 3) * (NDIM * 2) + ((lane & 7) ^ (((lane >> 4) & 1) << 2)) * 16;
    const char* xl0 = p.xl[ci][0]; const char* xl1 = p.xl[ci][1]; const char* xl2 = p.xl[ci][2]; const char* xl3 = p.xl[ci][3];
    unsigned r0 = p.cls[ci][0], rd = p.cls[ci][1], tpb = p.cls[ci][2], tps = p.cls[ci][3];
    {
        const unsigned zero = 0, fa = (unsigned)(size_t)(LAS char*)lds + L_FLAG + (lane & 15) * 4;
        asm volatile("ds_write_b32 %12, %13\n\ts_memrealtime %11\n\ts_waitcnt lgkmcnt(0)\n\ts_barrier"
                     : "+s"(wg), "+s"(me), "+s"(xl0), "+s"(xl1), "+s"(xl2), "+s"(xl3), "+s"(r0), "+s"(rd), "+s"(tpb), "+s"(tps), "+s"(ta), "=&s"(tb)
                     : "v"(fa), "v"(zero) : "memory");
    }
    TIME_BASE();
    const unsigned tp0 = tpb + jj * tps;
    if (grp == 1) {
        if (tp0 > T_BIAS / 2) wait_until(t0, tp0);
        const unsigned xlane = (16 * u + (lane >> 3)) * (KLOC * 2) + ((lane & 7) ^ (lane >> 3)) * 16 + w2 * 128;
        const char* wsrc = wg + (64 * w2 * NDIM + 64 * jj) * 2;
#pragma unroll
        for (int h = 0; h < 2; ++h) {
#pragma unroll
            for (int i = 0; i < 8; ++i)
                GLDS(wsrc + (size_t)(h * 128 + i * 8) * NDIM * 2 + wlane, lds + L_W + h * 16384 + w2 * 8192 + i * 1024);
#pragma unroll
            for (int q = 0; q < 4; ++q) {
                const char* xs = (q == 0 ? xl0 : q == 1 ? xl1 : q == 2 ? xl2 : xl3) + h * 256;
                GLDS(xs + xlane, lds + L_XL + q * 8192 + h * 4096 + w2 * 2048);
                GLDS(xs + 8 * KLOC * 2 + xlane, lds + L_XL + q * 8192 + h * 4096 + w2 * 2048 + 1024);
            }
        }
    }
    GAS unsigned* cnt = (GAS unsigned*)(me + CTRL_OFF + 4096) + (b * 8 + wave);
    GAS unsigned* tmo = (GAS unsigned*)(me + CTRL_OFF);
    const unsigned cval = __hip_atomic_load(cnt, __ATOMIC_RELAXED, __HIP_MEMORY_SCOPE_AGENT);
    const unsigned tval = __hip_atomic_load(tmo, __ATOMIC_RELAXED, __HIP_MEMORY_SCOPE_AGENT);
    char* dst0 = p.dst[ci][0]; char* dst1 = p.dst[ci][1]; char* dst2 = p.dst[ci][2]; char* dst3 = p.dst[ci][3];
    char* outp = p.out;
    unsigned prefm = p.pref, rlead = p.rlead, rblk = p.rblk, rl2 = p.rl2, gmin = p.gmin, gmax = p.gmax, gend = p.gend, tb0 = p.tb0;
    unsigned sprint = p.sprint, sprc = p.sprc, dtx = p.dt[ci * 8 + xcd], tlc = p.tl[grp], tl0 = p.tl[0], tl1 = p.tl[1];
    unsigned txu = p.tx[xslot], twc = p.tw[c];
    asm volatile("" : "+s"(dst0), "+s"(dst1), "+s"(dst2), "+s"(dst3), "+s"(outp), "+s"(prefm), "+s"(rlead), "+s"(rblk), "+s"(rl2),
                      "+s"(gmin), "+s"(gmax), "+s"(gend), "+s"(tb0), "+s"(sprint), "+s"(sprc), "+s"(dtx), "+s"(tlc), "+s"(tl0), "+s"(tl1),
                      "+s"(txu), "+s"(twc));
    txu >>= 16;
    twc >>= 16;
    if (grp == 1) {
        asm volatile("s_waitcnt vmcnt(18)" ::: "memory");
        flag_add(lds, F_LA, lane);
    }
    const char* xl_rd0 = lds + L_XL + (lane & 15) * 128 + (g4 ^ (lane & 7)) * 16;
    const char* xl_rd1 = lds + L_XL + (lane & 15) * 128 + ((4 + g4) ^ (lane & 7)) * 16;
    const char* w_rd = lds + L_W + (grp & 1) * 32768 + (8 * g4 + q4) * 128 + ((w2 ^ (q4 >> 1)) * 64) + pc * 16;
    const unsigned wroff = (c * 2 + w2) * 4096u + lane * 16u;
    const unsigned toff = (((isA ? (c >> 1) * RA_ + u : (c >> 1) * RB_ + (u - NA)) * 2 + (c & 1)) * 2 + w2) * 1024u + lane * 16u;
    u32x4 wr[4][4];
#pragma unroll
    for (int k = 0; k < 16; ++k) wr[k >> 2][k & 3] = u32x4{0u, 0u, 0u, 0u};
    WAIT_VM0();
    if (grp == 1) flag_add(lds, F_LB, lane);
    const unsigned call = __builtin_amdgcn_readfirstlane(cval) + 1;
    const unsigned bufi = call % 3;
    const unsigned bufoff = bufi * (isA ? ABUF : CBUF);
    bool bad = __builtin_amdgcn_readfirstlane(tval) != 0;
    if (lane == 0) __hip_atomic_store(cnt, call, __ATOMIC_RELAXED, __HIP_MEMORY_SCOPE_AGENT);
    char* cin = me + COFF + bufi * CBUF;
    auto rl = RSRC(cin, 10u * CBUF);
    auto give_up = [&]() -> bool {
        const unsigned el = now32() - t0;
        if (!bad && el < 10000u) return false;
        if (bad || el > DEADLINE_TICKS ||
            __builtin_amdgcn_readfirstlane(__hip_atomic_load(tmo, __ATOMIC_RELAXED, __HIP_MEMORY_SCOPE_AGENT)) != 0) {
            if (lane == 0) __hip_atomic_store(tmo, call, __ATOMIC_RELAXED, __HIP_MEMORY_SCOPE_AGENT);
            bad = true;
            return true;
        }
        __builtin_amdgcn_s_sleep(64);
        return false;
    };
    if (grp >= 2) {
        const unsigned src = wave - 4;
        auto load_w1 = [&]() {
            wait_until(t0, tp0 + tl1);
            const char* wsrc = wg + (64 * src * NDIM + 64 * (jj + 8)) * 2;
#pragma unroll
            for (int i = 0; i < 8; ++i) GLDS(wsrc + (size_t)i * 8 * NDIM * 2 + wlane, lds + L_W + 32768 + src * 8192 + i * 1024);
            WAIT_VM0();
            flag_add(lds, F_WL + 1, lane);
        };
        const bool xfirst = txu <= tp0 + tl0;
        if (!xfirst) load_w1();
        wait_until(t0, txu);
        u32x4 xr[8];
        for (;;) {
            unsigned mn = 0xffffffffu;
#pragma unroll
            for (int i = 0; i < 8; ++i)
                xr[i] = __builtin_amdgcn_raw_buffer_load_b128(rl, src * (3 * CBUF) + XIN + xslot * 8192u + i * 1024u + lane * 16u, 0, AUX_SYS);
#pragma unroll
            for (int i = 0; i < 8; ++i) mn = min(mn, min4(xr[i]));
            if (__all(mn != 0)) break;
            if (give_up()) {
#pragma unroll
                for (int i = 0; i < 8; ++i) xr[i] = u32x4{NAN2, NAN2, NAN2, NAN2};
                break;
            }
            __builtin_amdgcn_s_sleep(XSL);
        }
#pragma unroll
        for (int i = 0; i < 8; ++i) *(u32x4*)(lds + L_XR + src * 8192 + i * 1024 + lane * 16) = xr[i];
        WAIT_LGKM0();
        flag_add(lds, F_XR + src, lane);
        STAMP(3);
        if (xfirst) load_w1();
        flag_wait(lds, F_FREE + (grp & 1), 2);
        wait_until(t0, tlc);
        const char* wsrc = wg + (128 * w2 * NDIM + 64 * c) * 2;
#pragma unroll
        for (int i = 0; i < 16; ++i) GLDS(wsrc + (size_t)i * 8 * NDIM * 2 + wlane, lds + L_W + (grp & 1) * 32768 + w2 * 16384 + i * 1024);
        WAIT_VM0();
        flag_add(lds, F_WL + grp, lane);
    }
    const u32x4 rlw = {(unsigned)(size_t)cin, (unsigned)((size_t)cin >> 32) & 0xffffu, 10u * CBUF, 0x00020000u};
#define WR_ALL "+v"(wr[0][0]), "+v"(wr[0][1]), "+v"(wr[0][2]), "+v"(wr[0][3]), "+v"(wr[1][0]), "+v"(wr[1][1]), "+v"(wr[1][2]), "+v"(wr[1][3]), \
               "+v"(wr[2][0]), "+v"(wr[2][1]), "+v"(wr[2][2]), "+v"(wr[2][3]), "+v"(wr[3][0]), "+v"(wr[3][1]), "+v"(wr[3][2]), "+v"(wr[3][3])
    auto load_wr = [&](unsigned mask) {
        const unsigned s1 = 3 * CBUF, s2 = 6 * CBUF, s3 = 9 * CBUF;
#define LD4(a, b_, c_, d_, so) \
        "buffer_load_dwordx4 %" #a ", %16, %17, " so " offen sc0 sc1\n\t" \
        "buffer_load_dwordx4 %" #b_ ", %16, %17, " so " offen offset:1024 sc0 sc1\n\t" \
        "buffer_load_dwordx4 %" #c_ ", %16, %17, " so " offen offset:2048 sc0 sc1\n\t" \
        "buffer_load_dwordx4 %" #d_ ", %16, %17, " so " offen offset:3072 sc0 sc1\n\t"
        asm volatile("s_nop 4\n\t"
                     "s_bitcmp0_b32 %18, 0\n\ts_cbranch_scc1 1f\n\t" LD4(0, 1, 2, 3, "0")
                     "1:\n\ts_bitcmp0_b32 %18, 1\n\ts_cbranch_scc1 2f\n\t" LD4(4, 5, 6, 7, "%19")
                     "2:\n\ts_bitcmp0_b32 %18, 2\n\ts_cbranch_scc1 3f\n\t" LD4(8, 9, 10, 11, "%20")
                     "3:\n\ts_bitcmp0_b32 %18, 3\n\ts_cbranch_scc1 4f\n\t" LD4(12, 13, 14, 15, "%21")
                     "4:"
                     : WR_ALL : "v"(wroff), "s"(rlw), "s"(mask), "s"(s1), "s"(s2), "s"(s3) : "memory", "scc");
    };
    auto wait_wr = [&]() { asm volatile("s_waitcnt vmcnt(0)" : WR_ALL :: "memory"); };
    auto wmin = [&](int sc) { return min(min(min4(wr[sc][0]), min4(wr[sc][1])), min(min4(wr[sc][2]), min4(wr[sc][3]))); };
    const bool pref = twc <= max(prefm, now32() - t0 + 25u);
    if (pref) wait_until(t0, twc);
    load_wr(__builtin_amdgcn_readfirstlane(pref ? 15u : 0u));
    unsigned tres = r0 + (((c & ~1u) * (rd >> 16) + 2u * (isA ? u : u - NA) * (rd & 0xffffu)) >> 4);
    unsigned xr_a = (unsigned)(size_t)(LAS char*)lds + L_XR + lane * 16;
    unsigned fa = (unsigned)(size_t)(LAS char*)lds + L_FLAG + F_XR * 4 + (lane & 3) * 4;
    char* pacer0 = me + CTRL_OFF + 1024 + (ci * 8 + (xcd & sprint)) * 128;
    unsigned long long tmin = isA ? 0ull : t064 + tb0;
    asm volatile("" : "+s"(tres), "+s"(dtx), "+s"(pacer0), "+s"(tmin), "+v"(xr_a), "+v"(fa));
    if (grp >= 1) { flag_wait(lds, F_WL + grp, grp == 1 ? 4 : 2); flag_wait(lds, F_LB, 2); }
    flag_wait(lds, F_LA, 2);
    STAMP(1);

    f32x4 acc[4][2];
    unsigned wm[4];
    {
#pragma unroll
        for (int q = 0; q < 4; ++q) { acc[q][0] = f32x4{0.f, 0.f, 0.f, 0.f}; acc[q][1] = acc[q][0]; }
        u32x4 wf[2][2], xf[2][4];
        auto rdl = [&](int s) {
            wf[s & 1][0] = tr_frag(w_rd + s * 4096, 512);
            xf[s & 1][0] = *(const u32x4*)(((s & 1) ? xl_rd1 : xl_rd0) + (0 * 4 + (s >> 1)) * 2048);
            wf[s & 1][1] = tr_frag(w_rd + s * 4096 + 8, 512);
#pragma unroll
            for (int q = 1; q < 4; ++q)
                xf[s & 1][q] = *(const u32x4*)(((s & 1) ? xl_rd1 : xl_rd0) + (q * 4 + (s >> 1)) * 2048);
        };
        rdl(0);
        SB();
#pragma unroll
        for (int s = 0; s < 8; ++s) {
            if (s == 3) { flag_wait(lds, F_LB, 2); SB(); }
            if (s == 6) { wait_wr(); SB(); }
            if (s + 1 < 8) rdl(s + 1);
#pragma unroll
            for (int q = 0; q < 4; ++q) {
                acc[q][0] = MFMA(wf[s & 1][0], xf[s & 1][q], acc[q][0]);
                acc[q][1] = MFMA(wf[s & 1][1], xf[s & 1][q], acc[q][1]);
            }
            if (s >= 6) { wm[2 * (s - 6)] = wmin(2 * (s - 6)); wm[2 * (s - 6) + 1] = wmin(2 * (s - 6) + 1); }
            if (s < 6) {
                SGB(0x100, 2); SGB(0x8, 1); SGB(0x100, 2); SGB(0x8, 1); SGB(0x100, 1); SGB(0x8, 1); SGB(0x100, 1); SGB(0x8, 1);
                SGB(0x100, 1); SGB(0x8, 1); SGB(0x100, 1); SGB(0x8, 3);
            } else if (s == 6) {
                SGB(0x100, 2); SGB(0x8, 1); SGB(0x2, 2); SGB(0x100, 2); SGB(0x8, 1); SGB(0x2, 2); SGB(0x100, 1); SGB(0x8, 1); SGB(0x2, 2);
                SGB(0x100, 1); SGB(0x8, 1); SGB(0x2, 2); SGB(0x100, 1); SGB(0x8, 1); SGB(0x2, 2); SGB(0x100, 1); SGB(0x8, 1); SGB(0x2, 2);
                SGB(0x8, 1); SGB(0x2, 2); SGB(0x8, 1); SGB(0x2, 2);
            } else {
                SGB(0x8, 1); SGB(0x2, 2); SGB(0x8, 1); SGB(0x2, 2); SGB(0x8, 1); SGB(0x2, 2); SGB(0x8, 1); SGB(0x2, 2);
                SGB(0x8, 1); SGB(0x2, 2); SGB(0x8, 1); SGB(0x2, 2); SGB(0x8, 1); SGB(0x2, 2); SGB(0x8, 1); SGB(0x2, 2);
            }
            SB();
            if (s >= 6) asm volatile("" : "+v"(wm[2 * (s - 6)]), "+v"(wm[2 * (s - 6) + 1]));
        }
        if (grp < 2) flag_add(lds, F_FREE + grp, lane);
    }
    STAMP(2);
    unsigned long long slot = 0;
    auto reserve = [&](unsigned long long want) {
        unsigned vz = 0;
        asm volatile("" : "+v"(vz));
        GAS unsigned long long* pacer = (GAS unsigned long long*)(pacer0 + vz);
        if (want < tmin) want = tmin;
        if (lane == 0) {
            __hip_atomic_fetch_max(pacer, want << 8, __ATOMIC_RELAXED, __HIP_MEMORY_SCOPE_WORKGROUP);
            slot = __hip_atomic_fetch_add(pacer, (unsigned long long)dtx, __ATOMIC_RELAXED, __HIP_MEMORY_SCOPE_WORKGROUP);
        }
    };
    bool early = false;
    {
        unsigned done = 0;
        for (;;) {
            unsigned fl;
            asm volatile("ds_read_b32 %0, %1" : "=v"(fl) : "v"(fa) : "memory");
            unsigned wok = 0;
#pragma unroll
            for (int sc = 0; sc < 4; ++sc) if (__all(wm[sc] != 0)) wok |= 1u << sc;
            wok = __builtin_amdgcn_readfirstlane(wok);
            if (wok == 0) wait_until(t0, twc);
            load_wr(~wok & 15u);
            asm volatile("s_waitcnt lgkmcnt(0)" : "+v"(fl) :: "memory");
            const unsigned ready = wok & (unsigned)__builtin_amdgcn_ballot_w64(fl != 0) & ~done & 15u;
            if (ready) {
                u32x4 xf[8];
                const unsigned a0 = xr_a + __builtin_ctz(ready) * 8192;
#pragma unroll
                for (int k = 0; k < 8; ++k) xf[k] = *(const LAS u32x4*)(size_t)(a0 + k * 1024);
                SB();
                if ((ready | done) == 15u) {
                    STAMP(5);
                    const unsigned long long tn = __builtin_amdgcn_s_memrealtime();
                    const unsigned ahead = rlead + rblk * __builtin_popcount(ready);
                    early = (unsigned)tn - t0 + ahead >= tres;
                    if (early) reserve(tn + ahead);
                }
#pragma unroll
                for (int sc = 0; sc < 4; ++sc)
                    if ((ready >> sc) & 1) {
                        const unsigned an = xr_a + (__builtin_ctz((ready >> (sc + 1)) | 8u) + sc + 1 & 3) * 8192;
#pragma unroll
                        for (int k = 0; k < 8; ++k) {
                            acc[k >> 1][0] = MFMA(wr[sc][(k & 1) * 2], xf[k], acc[k >> 1][0]);
                            acc[k >> 1][1] = MFMA(wr[sc][(k & 1) * 2 + 1], xf[k], acc[k >> 1][1]);
                            if (sc < 3) xf[k] = *(const LAS u32x4*)(size_t)(an + k * 1024);
                            SB();
                        }
                    }
            }
            done |= ready;
            if (done == 15u) break;
            const bool gu = give_up();
            wait_wr();
            if (gu) break;
            if (wok == 15u) __builtin_amdgcn_s_sleep(1);
            else {
#pragma unroll
                for (int sc = 0; sc < 4; ++sc) wm[sc] = wmin(sc);
            }
        }
    }
    unsigned gate;
    {
        u32x4 d[4];
#pragma unroll
        for (int q = 0; q < 4; ++q)
#pragma unroll
            for (int k = 0; k < 4; ++k) d[q][k] = nz(pack2(acc[q][k >> 1][(2 * k) & 3], acc[q][k >> 1][(2 * k + 1) & 3]));
        if (bad) {
#pragma unroll
            for (int q = 0; q < 4; ++q) d[q] = u32x4{NAN2, NAN2, NAN2, NAN2};
        }
        auto push = [&](int q) {
            auto rs = RSRC((q == 0 ? dst0 : q == 1 ? dst1 : q == 2 ? dst2 : dst3) + bufoff, isA ? ABUF : CBUF - PBO);
            __builtin_amdgcn_raw_buffer_store_b128(d[q], rs, toff, 0, AUX_SYS);
        };
        STAMP(6);
        if (!early) {
            wait_until(t0, tres);
            bool again = true;
            if (c >= sprc) {
                unsigned vz = 0;
                asm volatile("" : "+v"(vz));
                if (lane == 0) slot = __hip_atomic_fetch_add((GAS unsigned long long*)(pacer0 + vz), (unsigned long long)dtx, __ATOMIC_RELAXED, __HIP_MEMORY_SCOPE_WORKGROUP);
                again = (unsigned)__builtin_amdgcn_readfirstlane((unsigned)(slot >> 8)) - now32() >= 4000u;
            }
            if (again) reserve(__builtin_amdgcn_s_memrealtime() + rl2);
        }
        asm volatile("" : "+v"(slot));
        gate = __builtin_amdgcn_readfirstlane((unsigned)(slot >> 8)) - t0;
        for (;;) {
            const unsigned tn = now32() - t0;
            if (gate - tn - 1u >= 4000u) { gate = tn; break; }
            __builtin_amdgcn_s_sleep(1);
        }
        push(1); push(2); push(3);
        if (!isA) push(0);
    }
    STAMP(7);
    const unsigned gst = isA ? 3 * ABUF : 3 * CBUF;
    auto rg = RSRC(me + (isA ? AOFF : COFF + PBO) + bufoff, isA ? 10u * ABUF : 10u * CBUF - PBO);
    const unsigned so0 = isA ? gst : 0u;
    GAS u32x4* optr = (GAS u32x4*)(outp + ((u * 16 + (lane & 15)) * NDIM + c * 64 + w2 * 32 + g4 * 8) * 2);
    u32x4 in[4];
    wait_until(t0, max(gate + gmin, min(gate + gmax, gend)));
    for (;;) {
        in[0] = __builtin_amdgcn_raw_buffer_load_b128(rg, toff, so0, AUX_SYS);
        in[1] = __builtin_amdgcn_raw_buffer_load_b128(rg, toff, gst, AUX_SYS);
        in[2] = __builtin_amdgcn_raw_buffer_load_b128(rg, toff, 2 * gst, AUX_SYS);
        in[3] = __builtin_amdgcn_raw_buffer_load_b128(rg, toff, 3 * gst, AUX_SYS);
        if (__all(min(min(min4(in[0]), min4(in[1])), min(min4(in[2]), min4(in[3]))) != 0)) break;
        if (give_up()) break;
    }
    STAMP(9);
    float y[8];
#pragma unroll
    for (int k = 0; k < 4; ++k) {
        y[2 * k] = isA ? acc[0][k >> 1][(2 * k) & 3] : lo_f(in[0][k]);
        y[2 * k + 1] = isA ? acc[0][k >> 1][(2 * k + 1) & 3] : hi_f(in[0][k]);
    }
#pragma unroll
    for (int q = 1; q < 4; ++q)
#pragma unroll
        for (int k = 0; k < 4; ++k) {
            y[2 * k] += lo_f(in[q][k]);
            y[2 * k + 1] += hi_f(in[q][k]);
        }
    u32x4 o = {pack2(relu(y[0]), relu(y[1])), pack2(relu(y[2]), relu(y[3])),
               pack2(relu(y[4]), relu(y[5])), pack2(relu(y[6]), relu(y[7]))};
    if (bad) o = u32x4{NAN2, NAN2, NAN2, NAN2};
    *optr = o;
}

extern "C" void kernel_launch(int rank, int nranks, ncclComm_t comm,
                              hipStream_t stream,
                              void* const* d_in, const int* in_sizes, int n_in,
                              void* d_out, int out_size,
                              const int* in_dtypes, int out_dtype,
                              void* const* d_out_all,
                              void* const* d_ws_all, size_t ws_bytes) {
    (void)comm; (void)in_dtypes; (void)out_dtype; (void)d_out_all;
    if (nranks != NR || (unsigned)rank >= (unsigned)NR || n_in < 2 || ws_bytes < WS_NEED ||
        in_sizes[0] != MDIM * KLOC || in_sizes[1] != KLOC * NDIM || out_size != MLOC * NDIM) {
        fprintf(stderr, "kernel_launch: unexpected problem shape\n");
        return;
    }
    Params p = {};
    const int g4 = rank & 4, gi = rank & 3;
    const char* x = (const char*)d_in[0];
    const size_t own = (size_t)MLOC * KLOC * 2;
    p.w = (const char*)d_in[1];
    p.out = (char*)d_out;
    p.me = (char*)d_ws_all[rank];
    p.xg[0] = x + g4 * own;
    p.xg[1] = x + (g4 ^ 4) * own;
    for (int q = 0; q < 4; ++q) {
        const int o = (gi + q) & 3;
        p.xl[0][q] = x + (g4 + o) * own;
        p.xl[1][q] = x + ((g4 ^ 4) + o) * own;
        p.dst[0][q] = (char*)d_ws_all[g4 + o] + AOFF + (size_t)(q ? 4 - q : 1) * 3 * ABUF;
        p.dst[1][q] = (char*)d_ws_all[(g4 ^ 4) + o] + COFF + (size_t)gi * 3 * CBUF + PBO;
        p.crs[q] = (char*)d_ws_all[(g4 ^ 4) + q] + COFF + (size_t)gi * 3 * CBUF;
    }
    p.tch[0] = p.me + COFF + 6 * CBUF; p.tch[1] = p.me + COFF + 12 * CBUF - 64; p.tch[2] = p.me + WS_DATA - 64;
    for (int q = 1; q < 4; ++q) p.tch[2 + q] = p.dst[0][q];
    for (int q = 0; q < 4; ++q) p.tch[6 + q] = p.crs[q];
    p.tch[10] = p.out; p.tch[11] = p.out + (1u << 20) - 64;
    p.ttch = 0;
    unsigned t_first = 60, unit16 = 220;
    const char* order = "XXWWXXWWXWXWXWXWXWXWWWXWXWXWXWXWXWXWWWWWWWWWWWWWWW";
    p.tz = 200; p.dz = 100; p.slead = 130; p.pref = 0;
    p.rlead = 15; p.rblk = 14; p.rl2 = 30; p.gmin = 50; p.gmax = 250; p.gend = 1430; p.tb0 = 740; p.sprint = 7; p.sprc = 99;
    p.tl[0] = 250; p.tl[1] = 145; p.tl[2] = 380; p.tl[3] = 440;
    unsigned da = 56, db = 56, lead = 40, leadmax = 110, tlb = 130, tlj = 10, nfast = 10, tla = 50, ra0 = 180, rb0 = 640;
    t_first += T_BIAS; p.tz += T_BIAS; p.gend += T_BIAS; p.tb0 += T_BIAS; p.tl[2] += T_BIAS; p.tl[3] += T_BIAS; ra0 += T_BIAS; rb0 += T_BIAS; tlb += T_BIAS;
    p.sub16 = unit16 / 4;
    const unsigned nwa = 8 * NA + NSPLIT, nwb = 128 - nwa;
    p.cls[0][0] = ra0; p.cls[0][1] = da | ((da * nwa / 8) << 16); p.cls[0][2] = tla; p.cls[0][3] = tlj;
    p.cls[1][0] = rb0; p.cls[1][1] = db | ((db * nwb / 8) << 16); p.cls[1][2] = tlb; p.cls[1][3] = 0;
    const int nxs = NSPLIT ? 17 : 16;
    int nx = 0, nw = 0; unsigned pos = 0;
    for (const char* s = order; *s; ++s) {
        if (*s == 'X' && nx < nxs) {
            const unsigned t = t_first + ((pos++ * unit16) >> 4);
            const int slot = !NSPLIT ? nx : nx < NA ? nx : nx == NA ? 16 : nx - 1;
            p.tx[slot] = t | ((t + ((slot < NA || slot == 16) ? lead : leadmax)) << 16); ++nx;
        } else if (*s == 'W' && nw < 32) {
            const unsigned t = t_first + ((pos++ * unit16) >> 4);
            p.tw[nw] = t | ((t + (nw < (int)nfast ? lead : leadmax)) << 16); ++nw;
        }
    }
    if (nx != nxs || nw != 32) { fprintf(stderr, "bad order\n"); return; }
    for (int xc = 0; xc < 8; ++xc) {
        unsigned ra = 0, rb = 0;
        for (int idx = 0; idx < 16; ++idx) {
            for (int x2 = 0; x2 < 8; ++x2) {
                if ((x2 & p.sprint) != (xc & p.sprint)) continue;
                const int u = (x2 >> 1) + 4 * (idx & 3), jj = (idx >> 2) * 2 + (x2 & 1);
                ((u < NA || (u == NA && jj < NSPLIT)) ? ra : rb)++;
            }
        }
        p.dt[xc] = ra ? da * nwa * 8 / ra : 0;
        p.dt[8 + xc] = rb ? db * nwb * 8 / rb : 0;
    }
    gemm_rs_kernel<<<256, 512, 0, stream>>>(p);
    HIP_CHECK(hipGetLastError());
}
```

```cpp
#include <hip/hip_runtime.h>
#include <rccl/rccl.h>
#include <cstdio>
#include <cstdint>

#define HIP_CHECK(expr) do { hipError_t _e = (expr); if (_e != hipSuccess) { \
    fprintf(stderr, "HIP error %s at %s:%d\n", hipGetErrorString(_e), __FILE__, __LINE__); \
    return; } } while (0)

typedef __attribute__((ext_vector_type(8))) __bf16 bf16x8;
typedef __attribute__((ext_vector_type(4))) __bf16 bf16x4;
typedef __attribute__((ext_vector_type(2))) __bf16 bf16x2;
typedef __attribute__((ext_vector_type(2))) float f32x2;
typedef __attribute__((ext_vector_type(4))) float f32x4;
typedef __attribute__((ext_vector_type(4))) unsigned u32x4;
typedef __attribute__((ext_vector_type(2))) unsigned u32x2;
#define LAS __attribute__((address_space(3)))
#define GAS __attribute__((address_space(1)))

constexpr int NR = 8;
constexpr int MDIM = 2048, NDIM = 2048, KLOC = 256, MLOC = 256;
constexpr unsigned NA = 9;
constexpr unsigned NSPLIT = 4;
constexpr unsigned NEARLY = 4;
constexpr unsigned SCOUT = 5;

constexpr unsigned PACER_OFF = 1024, CNT_OFF = 4096;
constexpr unsigned RA_ = NA + 1, RB_ = 16 - NA;
constexpr unsigned XIN = 32u * 8192u;
constexpr unsigned PBO = XIN + 17u * 8192u;
constexpr unsigned CBUF = PBO + RB_ * 65536u;
constexpr unsigned ABUF = RA_ * 65536u;
constexpr unsigned COFF = 64u << 10;
constexpr unsigned AOFF = COFF + 12u * CBUF - 3u * ABUF;
constexpr unsigned WS_DATA = AOFF + 12u * ABUF;

constexpr unsigned DEADLINE_TICKS = 100000000u;
constexpr int AUX_SYS = 17;
constexpr int L_W = 0, L_XL = 65536, L_XR = 98304, L_FLAG = 131072, L_TOTAL = 131072 + 64;
constexpr unsigned NAN2 = 0x7fc07fc0u;

constexpr unsigned T_BIAS = 100;
constexpr unsigned T_FIRST = T_BIAS + 60;
constexpr unsigned UNIT16 = 220;
constexpr unsigned T_B0 = T_BIAS + 736;
constexpr unsigned TILE16 = 56;
constexpr unsigned T_ZERO = T_BIAS + 200, D_ZERO = 100;
constexpr unsigned S_LEAD = 130;
constexpr unsigned LEAD_FAST = 40, LEAD_LAZY = 110, NFAST = 10;
constexpr unsigned R_A0 = T_BIAS + 180, R_B0 = T_BIAS + 640;
constexpr unsigned R_LEAD = 15, R_BLOCK = 14, R_LATE = 30;
constexpr unsigned G_MIN = 50, G_MAX = 250, G_END = T_BIAS + 1430;
constexpr unsigned L_A0 = 50, L_AJ = 10, L_B0 = T_BIAS + 130;
constexpr unsigned L_XFIRST = 250, L_W1 = 145, L_W2 = T_BIAS + 380, L_W3 = T_BIAS + 440;
constexpr unsigned NWA = 8 * NA + NSPLIT, NWB = 128 - NWA;

struct Params {
    const char* w;
    char* me;
    const char* xg[2];
    const char* xl[2][4];
    char* dst[2][4];
    char* crs[4];
    char* out;
    unsigned dt[16];
    unsigned tx[18];
    unsigned tw[32];
    char* tch[12];
};

__device__ __forceinline__ unsigned pack2(float a, float b) {
    f32x2 v = {a, b};
    return __builtin_bit_cast(unsigned, __builtin_convertvector(v, bf16x2));
}
__device__ __forceinline__ unsigned nz(unsigned d) { return d | ((d - 1u) & 0x80000000u); }
__device__ __forceinline__ u32x4 nz4(u32x4 d) { return u32x4{nz(d[0]), nz(d[1]), nz(d[2]), nz(d[3])}; }
__device__ __forceinline__ float lo_f(unsigned d) { return __builtin_bit_cast(float, d << 16); }
__device__ __forceinline__ float hi_f(unsigned d) { return __builtin_bit_cast(float, d & 0xffff0000u); }
__device__ __forceinline__ float relu(float y) { return y < 0.f ? 0.f : y; }
__device__ __forceinline__ unsigned min4(u32x4 v) { return min(min(v[0], v[1]), min(v[2], v[3])); }
__device__ __forceinline__ unsigned now32() { return (unsigned)__builtin_amdgcn_s_memrealtime(); }
__device__ __forceinline__ void wait_until(unsigned t0, unsigned d) {
    while (now32() - t0 < d) __builtin_amdgcn_s_sleep(1);
}
__device__ __forceinline__ u32x4 tr_frag(const char* a, int hi_off) {
    u32x2 lo = __builtin_bit_cast(u32x2, __builtin_amdgcn_ds_read_tr16_b64_v4bf16((LAS bf16x4*)a));
    u32x2 hi = __builtin_bit_cast(u32x2, __builtin_amdgcn_ds_read_tr16_b64_v4bf16((LAS bf16x4*)(a + hi_off)));
    return u32x4{lo[0], lo[1], hi[0], hi[1]};
}
#define MFMA(a, bb, c) __builtin_amdgcn_mfma_f32_16x16x32_bf16(__builtin_bit_cast(bf16x8, a), __builtin_bit_cast(bf16x8, bb), c, 0, 0, 0)
#define WAIT_VM0() __builtin_amdgcn_s_waitcnt(0x0F70)
#define WAIT_LGKM0() __builtin_amdgcn_s_waitcnt(0xC07F)
#define SGB(mask, n) __builtin_amdgcn_sched_group_barrier(mask, n, 0)
#define SB() __builtin_amdgcn_sched_barrier(0)
#define RSRC(base, bytes) __builtin_amdgcn_make_buffer_rsrc((base), 0, (int)(bytes), 0x00020000)
#define GLDS(src, dstlds) __builtin_amdgcn_global_load_lds((const GAS void*)(src), (LAS void*)(dstlds), 16, 0, 0)

__device__ __forceinline__ void flag_add(char* lds, int f, unsigned lane) {
    const unsigned a = (unsigned)(size_t)(LAS char*)lds + L_FLAG + f * 4, one = 1;
    if (lane == 0) asm volatile("ds_add_u32 %0, %1" :: "v"(a), "v"(one) : "memory");
}
__device__ __forceinline__ void flag_wait(char* lds, int f, unsigned n) {
    const unsigned a = (unsigned)(size_t)(LAS char*)lds + L_FLAG + f * 4;
    for (;;) {
        unsigned v;
        asm volatile("ds_read_b32 %0, %1\n\ts_waitcnt lgkmcnt(0)" : "=&v"(v) : "v"(a) : "memory");
        if ((unsigned)__builtin_amdgcn_readfirstlane(v) >= n) break;
        __builtin_amdgcn_s_sleep(1);
    }
}
enum { F_XR = 0  , F_LA = 4, F_LB = 5, F_WL = 6  , F_FREE = 10   };

__global__ __launch_bounds__(512, 2) void gemm_rs_kernel(Params p) {
    __shared__ __attribute__((aligned(16))) char lds[L_TOTAL];
    unsigned long long ta, tb;
    asm volatile("s_memrealtime %0" : "=s"(ta));
    SB();
    const char* wg = p.w; char* me = p.me; const char* xg0 = p.xg[0]; const char* xg1 = p.xg[1];
#define TIME_BASE() \
    const unsigned long long t064 = tb + min((unsigned)(tb - ta) >> 1, T_BIAS) - T_BIAS; \
    const unsigned t0 = (unsigned)t064
    const unsigned lane = threadIdx.x & 63;
    const unsigned wave = __builtin_amdgcn_readfirstlane(threadIdx.x >> 6);
    const unsigned b = blockIdx.x;
    const unsigned xcd = b & 7, idx = b >> 4;
    const unsigned g4 = lane >> 4, q4 = (lane >> 2) & 3, pc = lane & 3;

    if (__builtin_expect((b & 8) == 0, 1)) {
        const bool isW = wave < 2;
        const unsigned c = 2 * idx + (wave & 1), ss = xcd;
        const unsigned peer = xcd >> 1, xo = (xcd & 1) * 2 + (wave & 1);
        const unsigned xslot = wave < 4 ? idx : 16, u = wave < 4 ? idx : NA;
        const bool xa = idx < NA || wave >= 4;
        const unsigned mw = 0u - (unsigned)isW;
        const unsigned soff = (mw & ((32 * ss * NDIM + 64 * c) * 2)) | (~mw & (((xo * MLOC + 16 * u) * KLOC + peer * 64) * 2));
        const unsigned voff = (mw & ((lane >> 3) * (NDIM * 2) + (lane & 7) * 16)) | (~mw & ((lane & 15) * (KLOC * 2) + g4 * 16));
        const unsigned st1 = (mw & (8 * NDIM * 2)) | (~mw & 64), st2 = mw & (16 * NDIM * 2);
        const char* sbase = (isW ? wg : xa ? xg1 : xg0) + soff;
        asm volatile("s_memrealtime %0" : "=s"(tb), "+s"(sbase));
        u32x4 v[4];
        auto load = [&]() {
            v[0] = *(const GAS u32x4*)(sbase + voff);
            v[1] = *(const GAS u32x4*)(sbase + st1 + voff);
            v[2] = *(const GAS u32x4*)(sbase + st2 + voff);
            v[3] = *(const GAS u32x4*)(sbase + st1 + st2 + voff);
        };
        const bool fast = idx < NEARLY && wave < 4;
        if (fast) load();
        asm volatile("s_waitcnt lgkmcnt(0)" : "+s"(ta), "+s"(tb) :: "memory");
        TIME_BASE();
        char* pws = p.crs[peer];
        if (wave == 4 && idx == SCOUT) {
            unsigned sink = __hip_atomic_load((const GAS unsigned*)me, __ATOMIC_RELAXED, __HIP_MEMORY_SCOPE_AGENT);
            sink += __hip_atomic_load((const GAS unsigned*)(pws + (lane & 1) * (3 * CBUF - 64)), __ATOMIC_RELAXED, __HIP_MEMORY_SCOPE_SYSTEM);
            asm volatile("" :: "v"(sink));
#pragma unroll 1
            for (int i = 0; i < 12; ++i) {
                const unsigned len = i < 3 ? 0u : i < 6 ? 3 * ABUF - 64 : i < 10 ? 3 * CBUF - 64 : 0u;
                const unsigned t_ = __hip_atomic_load((const GAS unsigned*)(p.tch[i] + (lane & 1) * len), __ATOMIC_RELAXED, __HIP_MEMORY_SCOPE_SYSTEM);
                asm volatile("" :: "v"(t_));
            }
        }
        GAS unsigned* cnt = (GAS unsigned*)(me + CNT_OFF) + (b * 8 + wave);
        const unsigned cval = __hip_atomic_load(cnt, __ATOMIC_RELAXED, __HIP_MEMORY_SCOPE_AGENT);
        const unsigned tsend = (isW ? p.tw[c] : p.tx[xslot]) & 0xffffu;
        if (idx == 15 && wave == 1 && lane == 0)
            __hip_atomic_fetch_max((GAS unsigned long long*)(me + PACER_OFF + (8 + xcd) * 128), (t064 + T_B0) << 8,
                                   __ATOMIC_RELAXED, __HIP_MEMORY_SCOPE_WORKGROUP);
        if (wave < 4 || (wave < 6 && idx == 8)) {
            if (!fast) {
                wait_until(t0, tsend - S_LEAD);
                load();
            }
            const unsigned when = tsend + (((isW ? (ss & 1) * 2 : xo) * (UNIT16 / 4)) >> 4);
            if (isW) {
                char* sl = lds + wave * 4096;
#pragma unroll
                for (int q = 0; q < 4; ++q) *(u32x4*)(sl + q * 1024 + lane * 16) = v[q];
                const char* base = sl + (8 * g4 + q4) * 128 + pc * 16;
                v[0] = tr_frag(base, 512); v[1] = tr_frag(base + 8, 512);
                v[2] = tr_frag(base + 64, 512); v[3] = tr_frag(base + 72, 512);
            }
            const unsigned dst = (isW ? c * 8192u + (ss & 1) * 2048u : XIN + (xslot * 4 + ((xo - peer) & 3)) * 2048u) + lane * 16u;
#pragma unroll
            for (int q = 0; q < 4; ++q) v[q] = nz4(v[q]);
            auto rp = RSRC(pws + ((__builtin_amdgcn_readfirstlane(cval) + 1u) % 3u) * CBUF, PBO);
            wait_until(t0, when);
            __builtin_amdgcn_raw_buffer_store_b128(v[0], rp, dst, 0, AUX_SYS);
            __builtin_amdgcn_raw_buffer_store_b128(v[1], rp, dst + 1024, 0, AUX_SYS);
            if (isW) {
                __builtin_amdgcn_raw_buffer_store_b128(v[2], rp, dst + 4096, 0, AUX_SYS);
                __builtin_amdgcn_raw_buffer_store_b128(v[3], rp, dst + 4096 + 1024, 0, AUX_SYS);
            }
        }
        unsigned cv2 = cval;
        asm volatile("" : "+v"(cv2));
        const unsigned call = __builtin_amdgcn_readfirstlane(cv2) + 1;
        if (lane == 0) __hip_atomic_store(cnt, call, __ATOMIC_RELAXED, __HIP_MEMORY_SCOPE_AGENT);
        if (wave >= 4) {
            auto rz = RSRC(me, WS_DATA);
            const u32x4 z = {0u, 0u, 0u, 0u};
            const unsigned zb = (call + 2) % 3, zid = (idx * 8 + xcd) * 4 + (wave - 4);
            const unsigned zph = ((idx * 4 + (wave - 4)) * D_ZERO) >> 6;
            constexpr unsigned NCP = CBUF / 1024, NAP = ABUF / 1024;
            static_assert(4 * NCP + 3 * NAP <= 11 * 512, "re-arm loop too short");
#pragma unroll 1
            for (unsigned q = 0; q < 11; ++q) {
                const unsigned id = q * 512 + zid;
                if (id >= 4 * NCP + 3 * NAP) break;
                const unsigned i2 = id - 4 * NCP;
                const unsigned off = id < 4 * NCP ? COFF + ((id / NCP) * 3 + zb) * CBUF + (id % NCP) * 1024u
                                                  : AOFF + ((i2 / NAP + 1) * 3 + zb) * ABUF + (i2 % NAP) * 1024u;
                wait_until(t0, T_ZERO + zph + q * D_ZERO);
                __builtin_amdgcn_raw_buffer_store_b128(z, rz, off + lane * 16u, 0, AUX_SYS);
            }
        }
        return;
    }

    const unsigned u = (xcd >> 1) + 4 * (idx & 3);
    const unsigned jj = (idx >> 2) * 2 + (xcd & 1);
    const unsigned grp = wave >> 1, w2 = wave & 1;
    const unsigned c = jj + 8 * grp;
    const bool split = u == NA && jj < NSPLIT;
    const bool isA = u < NA || split;
    const unsigned xslot = split ? 16 : u;
    const unsigned ci = isA ? 0 : 1;
    const unsigned wlane = (lane >> 3) * (NDIM * 2) + ((lane & 7) ^ (((lane >> 4) & 1) << 2)) * 16;
    const char* xl0 = p.xl[ci][0]; const char* xl1 = p.xl[ci][1]; const char* xl2 = p.xl[ci][2]; const char* xl3 = p.xl[ci][3];
    {
        const unsigned zero = 0, fa = (unsigned)(size_t)(LAS char*)lds + L_FLAG + (lane & 15) * 4;
        asm volatile("ds_write_b32 %8, %9\n\ts_memrealtime %7\n\ts_waitcnt lgkmcnt(0)\n\ts_barrier"
                     : "+s"(wg), "+s"(me), "+s"(xl0), "+s"(xl1), "+s"(xl2), "+s"(xl3), "+s"(ta), "=&s"(tb)
                     : "v"(fa), "v"(zero) : "memory");
    }
    TIME_BASE();
    const unsigned tp0 = isA ? L_A0 + jj * L_AJ : L_B0;
    if (grp == 1) {
        if (tp0 > L_A0) wait_until(t0, tp0);
        const unsigned xlane = (16 * u + (lane >> 3)) * (KLOC * 2) + ((lane & 7) ^ (lane >> 3)) * 16 + w2 * 128;
        const char* wsrc = wg + (64 * w2 * NDIM + 64 * jj) * 2;
#pragma unroll
        for (int h = 0; h < 2; ++h) {
#pragma unroll
            for (int i = 0; i < 8; ++i)
                GLDS(wsrc + (size_t)(h * 128 + i * 8) * NDIM * 2 + wlane, lds + L_W + h * 16384 + w2 * 8192 + i * 1024);
#pragma unroll
            for (int q = 0; q < 4; ++q) {
                const char* xs = (q == 0 ? xl0 : q == 1 ? xl1 : q == 2 ? xl2 : xl3) + h * 256;
                GLDS(xs + xlane, lds + L_XL + q * 8192 + h * 4096 + w2 * 2048);
                GLDS(xs + 8 * KLOC * 2 + xlane, lds + L_XL + q * 8192 + h * 4096 + w2 * 2048 + 1024);
            }
        }
    }
    GAS unsigned* cnt = (GAS unsigned*)(me + CNT_OFF) + (b * 8 + wave);
    GAS unsigned* tmo = (GAS unsigned*)me;
    const unsigned cval = __hip_atomic_load(cnt, __ATOMIC_RELAXED, __HIP_MEMORY_SCOPE_AGENT);
    const unsigned tval = __hip_atomic_load(tmo, __ATOMIC_RELAXED, __HIP_MEMORY_SCOPE_AGENT);
    char* dst0 = p.dst[ci][0]; char* dst1 = p.dst[ci][1]; char* dst2 = p.dst[ci][2]; char* dst3 = p.dst[ci][3];
    char* outp = p.out;
    unsigned dtx = p.dt[ci * 8 + xcd], txu = p.tx[xslot], twc = p.tw[c];
    asm volatile("" : "+s"(dst0), "+s"(dst1), "+s"(dst2), "+s"(dst3), "+s"(outp), "+s"(dtx), "+s"(txu), "+s"(twc));
    txu >>= 16;
    twc >>= 16;
    if (grp == 1) {
        asm volatile("s_waitcnt vmcnt(18)" ::: "memory");
        flag_add(lds, F_LA, lane);
    }
    const char* xl_rd0 = lds + L_XL + (lane & 15) * 128 + (g4 ^ (lane & 7)) * 16;
    const char* xl_rd1 = lds + L_XL + (lane & 15) * 128 + ((4 + g4) ^ (lane & 7)) * 16;
    const char* w_rd = lds + L_W + (grp & 1) * 32768 + (8 * g4 + q4) * 128 + ((w2 ^ (q4 >> 1)) * 64) + pc * 16;
    const unsigned wroff = (c * 2 + w2) * 4096u + lane * 16u;
    const unsigned toff = (((isA ? (c >> 1) * RA_ + u : (c >> 1) * RB_ + (u - NA)) * 2 + (c & 1)) * 2 + w2) * 1024u + lane * 16u;
    u32x4 wr[4][4];
#pragma unroll
    for (int k = 0; k < 16; ++k) wr[k >> 2][k & 3] = u32x4{0u, 0u, 0u, 0u};
    WAIT_VM0();
    if (grp == 1) flag_add(lds, F_LB, lane);
    const unsigned call = __builtin_amdgcn_readfirstlane(cval) + 1;
    const unsigned bufi = call % 3;
    const unsigned bufoff = bufi * (isA ? ABUF : CBUF);
    bool bad = __builtin_amdgcn_readfirstlane(tval) != 0;
    if (lane == 0) __hip_atomic_store(cnt, call, __ATOMIC_RELAXED, __HIP_MEMORY_SCOPE_AGENT);
    char* cin = me + COFF + bufi * CBUF;
    auto rl = RSRC(cin, 10u * CBUF);
    auto give_up = [&]() -> bool {
        const unsigned el = now32() - t0;
        if (!bad && el < 10000u) return false;
        if (bad || el > DEADLINE_TICKS ||
            __builtin_amdgcn_readfirstlane(__hip_atomic_load(tmo, __ATOMIC_RELAXED, __HIP_MEMORY_SCOPE_AGENT)) != 0) {
            if (lane == 0) __hip_atomic_store(tmo, call, __ATOMIC_RELAXED, __HIP_MEMORY_SCOPE_AGENT);
            bad = true;
            return true;
        }
        __builtin_amdgcn_s_sleep(64);
        return false;
    };
    if (grp >= 2) {
        const unsigned src = wave - 4;
        auto load_w1 = [&]() {
            wait_until(t0, tp0 + L_W1);
            const char* wsrc = wg + (64 * src * NDIM + 64 * (jj + 8)) * 2;
#pragma unroll
            for (int i = 0; i < 8; ++i) GLDS(wsrc + (size_t)i * 8 * NDIM * 2 + wlane, lds + L_W + 32768 + src * 8192 + i * 1024);
            WAIT_VM0();
            flag_add(lds, F_WL + 1, lane);
        };
        const bool xfirst = txu <= tp0 + L_XFIRST;
        if (!xfirst) load_w1();
        wait_until(t0, txu);
        const unsigned xoff = src * (3 * CBUF) + XIN + xslot * 8192u;
        u32x4 xr[8];
        for (;;) {
            unsigned mn = 0xffffffffu;
#pragma unroll
            for (int i = 0; i < 8; ++i) xr[i] = __builtin_amdgcn_raw_buffer_load_b128(rl, xoff + i * 1024u + lane * 16u, 0, AUX_SYS);
#pragma unroll
            for (int i = 0; i < 8; ++i) mn = min(mn, min4(xr[i]));
            if (__all(mn != 0)) break;
            if (give_up()) {
#pragma unroll
                for (int i = 0; i < 8; ++i) xr[i] = u32x4{NAN2, NAN2, NAN2, NAN2};
                break;
            }
            for (;;) {
                __builtin_amdgcn_s_sleep(2);
                const unsigned cv = __builtin_amdgcn_raw_buffer_load_b32(rl, xoff + (lane & 7) * 1024u + 1020u, 0, AUX_SYS);
                if (__all(cv != 0) || give_up()) break;
            }
        }
#pragma unroll
        for (int i = 0; i < 8; ++i) *(u32x4*)(lds + L_XR + src * 8192 + i * 1024 + lane * 16) = xr[i];
        WAIT_LGKM0();
        flag_add(lds, F_XR + src, lane);
        if (xfirst) load_w1();
        flag_wait(lds, F_FREE + (grp & 1), 2);
        wait_until(t0, grp == 2 ? L_W2 : L_W3);
        const char* wsrc = wg + (128 * w2 * NDIM + 64 * c) * 2;
#pragma unroll
        for (int i = 0; i < 16; ++i) GLDS(wsrc + (size_t)i * 8 * NDIM * 2 + wlane, lds + L_W + (grp & 1) * 32768 + w2 * 16384 + i * 1024);
        WAIT_VM0();
        flag_add(lds, F_WL + grp, lane);
    }
    const u32x4 rlw = {(unsigned)(size_t)cin, (unsigned)((size_t)cin >> 32) & 0xffffu, 10u * CBUF, 0x00020000u};
#define WR_ALL "+v"(wr[0][0]), "+v"(wr[0][1]), "+v"(wr[0][2]), "+v"(wr[0][3]), "+v"(wr[1][0]), "+v"(wr[1][1]), "+v"(wr[1][2]), "+v"(wr[1][3]), \
               "+v"(wr[2][0]), "+v"(wr[2][1]), "+v"(wr[2][2]), "+v"(wr[2][3]), "+v"(wr[3][0]), "+v"(wr[3][1]), "+v"(wr[3][2]), "+v"(wr[3][3])
    auto load_wr = [&](unsigned mask) {
        const unsigned s1 = 3 * CBUF, s2 = 6 * CBUF, s3 = 9 * CBUF;
#define LD4(a, b_, c_, d_, so) \
        "buffer_load_dwordx4 %" #a ", %16, %17, " so " offen sc0 sc1\n\t" \
        "buffer_load_dwordx4 %" #b_ ", %16, %17, " so " offen offset:1024 sc0 sc1\n\t" \
        "buffer_load_dwordx4 %" #c_ ", %16, %17, " so " offen offset:2048 sc0 sc1\n\t" \
        "buffer_load_dwordx4 %" #d_ ", %16, %17, " so " offen offset:3072 sc0 sc1\n\t"
        asm volatile("s_nop 4\n\t"
                     "s_bitcmp0_b32 %18, 0\n\ts_cbranch_scc1 1f\n\t" LD4(0, 1, 2, 3, "0")
                     "1:\n\ts_bitcmp0_b32 %18, 1\n\ts_cbranch_scc1 2f\n\t" LD4(4, 5, 6, 7, "%19")
                     "2:\n\ts_bitcmp0_b32 %18, 2\n\ts_cbranch_scc1 3f\n\t" LD4(8, 9, 10, 11, "%20")
                     "3:\n\ts_bitcmp0_b32 %18, 3\n\ts_cbranch_scc1 4f\n\t" LD4(12, 13, 14, 15, "%21")
                     "4:"
                     : WR_ALL : "v"(wroff), "s"(rlw), "s"(mask), "s"(s1), "s"(s2), "s"(s3) : "memory", "scc");
    };
    auto wait_wr = [&]() { asm volatile("s_waitcnt vmcnt(0)" : WR_ALL :: "memory"); };
    auto wmin = [&](int sc) { return min(min(min4(wr[sc][0]), min4(wr[sc][1])), min(min4(wr[sc][2]), min4(wr[sc][3]))); };
    const bool pref = twc <= now32() - t0 + 25u;
    if (pref) wait_until(t0, twc);
    load_wr(__builtin_amdgcn_readfirstlane(pref ? 15u : 0u));
    unsigned tres = isA ? R_A0 + (((c & ~1u) * (TILE16 * NWA / 8) + 2u * u * TILE16) >> 4)
                        : R_B0 + (((c & ~1u) * (TILE16 * NWB / 8) + 2u * (u - NA) * TILE16) >> 4);
    unsigned xr_a = (unsigned)(size_t)(LAS char*)lds + L_XR + lane * 16;
    unsigned fa = (unsigned)(size_t)(LAS char*)lds + L_FLAG + F_XR * 4 + (lane & 3) * 4;
    char* pacer0 = me + PACER_OFF + (ci * 8 + xcd) * 128;
    asm volatile("" : "+s"(tres), "+s"(pacer0), "+v"(xr_a), "+v"(fa));
    if (grp >= 1) { flag_wait(lds, F_WL + grp, grp == 1 ? 4 : 2); flag_wait(lds, F_LB, 2); }
    flag_wait(lds, F_LA, 2);

    f32x4 acc[4][2];
    unsigned wm[4];
    {
#pragma unroll
        for (int q = 0; q < 4; ++q) { acc[q][0] = f32x4{0.f, 0.f, 0.f, 0.f}; acc[q][1] = acc[q][0]; }
        u32x4 wf[2][2], xf[2][4];
        auto rdl = [&](int s) {
            wf[s & 1][0] = tr_frag(w_rd + s * 4096, 512);
            xf[s & 1][0] = *(const u32x4*)(((s & 1) ? xl_rd1 : xl_rd0) + (0 * 4 + (s >> 1)) * 2048);
            wf[s & 1][1] = tr_frag(w_rd + s * 4096 + 8, 512);
#pragma unroll
            for (int q = 1; q < 4; ++q)
                xf[s & 1][q] = *(const u32x4*)(((s & 1) ? xl_rd1 : xl_rd0) + (q * 4 + (s >> 1)) * 2048);
        };
        rdl(0);
        SB();
#pragma unroll
        for (int s = 0; s < 8; ++s) {
            if (s == 3) { flag_wait(lds, F_LB, 2); SB(); }
            if (s == 6) { wait_wr(); SB(); }
            if (s + 1 < 8) rdl(s + 1);
#pragma unroll
            for (int q = 0; q < 4; ++q) {
                acc[q][0] = MFMA(wf[s & 1][0], xf[s & 1][q], acc[q][0]);
                acc[q][1] = MFMA(wf[s & 1][1], xf[s & 1][q], acc[q][1]);
            }
            if (s >= 6) { wm[2 * (s - 6)] = wmin(2 * (s - 6)); wm[2 * (s - 6) + 1] = wmin(2 * (s - 6) + 1); }
            if (s < 6) {
                SGB(0x100, 2); SGB(0x8, 1); SGB(0x100, 2); SGB(0x8, 1); SGB(0x100, 1); SGB(0x8, 1); SGB(0x100, 1); SGB(0x8, 1);
                SGB(0x100, 1); SGB(0x8, 1); SGB(0x100, 1); SGB(0x8, 3);
            } else if (s == 6) {
                SGB(0x100, 2); SGB(0x8, 1); SGB(0x2, 2); SGB(0x100, 2); SGB(0x8, 1); SGB(0x2, 2); SGB(0x100, 1); SGB(0x8, 1); SGB(0x2, 2);
                SGB(0x100, 1); SGB(0x8, 1); SGB(0x2, 2); SGB(0x100, 1); SGB(0x8, 1); SGB(0x2, 2); SGB(0x100, 1); SGB(0x8, 1); SGB(0x2, 2);
                SGB(0x8, 1); SGB(0x2, 2); SGB(0x8, 1); SGB(0x2, 2);
            } else {
                SGB(0x8, 1); SGB(0x2, 2); SGB(0x8, 1); SGB(0x2, 2); SGB(0x8, 1); SGB(0x2, 2); SGB(0x8, 1); SGB(0x2, 2);
                SGB(0x8, 1); SGB(0x2, 2); SGB(0x8, 1); SGB(0x2, 2); SGB(0x8, 1); SGB(0x2, 2); SGB(0x8, 1); SGB(0x2, 2);
            }
            SB();
            if (s >= 6) asm volatile("" : "+v"(wm[2 * (s - 6)]), "+v"(wm[2 * (s - 6) + 1]));
        }
        if (grp < 2) flag_add(lds, F_FREE + grp, lane);
    }
    unsigned long long slot = 0;
    auto reserve = [&](unsigned long long want) {
        unsigned vz = 0;
        asm volatile("" : "+v"(vz));
        GAS unsigned long long* pacer = (GAS unsigned long long*)(pacer0 + vz);
        if (lane == 0) {
            __hip_atomic_fetch_max(pacer, want << 8, __ATOMIC_RELAXED, __HIP_MEMORY_SCOPE_WORKGROUP);
            slot = __hip_atomic_fetch_add(pacer, (unsigned long long)dtx, __ATOMIC_RELAXED, __HIP_MEMORY_SCOPE_WORKGROUP);
        }
    };
    bool early = false;
    {
        unsigned done = 0;
        bool tried = pref;
        for (;;) {
            unsigned fl;
            asm volatile("ds_read_b32 %0, %1" : "=v"(fl) : "v"(fa) : "memory");
            unsigned wok = 0;
#pragma unroll
            for (int sc = 0; sc < 4; ++sc) if (__all(wm[sc] != 0)) wok |= 1u << sc;
            wok = __builtin_amdgcn_readfirstlane(wok);
            if (wok == 0) wait_until(t0, twc);
            unsigned want = ~wok & 15u;
            if (want && tried) {
                const unsigned cv = __builtin_amdgcn_raw_buffer_load_b32(rl, q4 * (3 * CBUF) + (c * 2 + w2) * 4096u + pc * 1024u + 1020u, 0, AUX_SYS);
                const unsigned zb_ = (unsigned)__builtin_amdgcn_ballot_w64(cv == 0);
                want &= ((zb_ & 0xfu) ? 0u : 1u) | ((zb_ & 0xf0u) ? 0u : 2u) | ((zb_ & 0xf00u) ? 0u : 4u) | ((zb_ & 0xf000u) ? 0u : 8u);
                if (!want) __builtin_amdgcn_s_sleep(2);
            }
            tried = true;
            load_wr(want);
            asm volatile("s_waitcnt lgkmcnt(0)" : "+v"(fl) :: "memory");
            const unsigned ready = wok & (unsigned)__builtin_amdgcn_ballot_w64(fl != 0) & ~done & 15u;
            if (ready) {
                u32x4 xf[8];
                const unsigned a0 = xr_a + __builtin_ctz(ready) * 8192;
#pragma unroll
                for (int k = 0; k < 8; ++k) xf[k] = *(const LAS u32x4*)(size_t)(a0 + k * 1024);
                SB();
                if ((ready | done) == 15u) {
                    const unsigned long long tn = __builtin_amdgcn_s_memrealtime();
                    const unsigned ahead = R_LEAD + R_BLOCK * __builtin_popcount(ready);
                    early = (unsigned)tn - t0 + ahead >= tres;
                    if (early) reserve(tn + ahead);
                }
#pragma unroll
                for (int sc = 0; sc < 4; ++sc)
                    if ((ready >> sc) & 1) {
                        const unsigned an = xr_a + ((__builtin_ctz((ready >> (sc + 1)) | 8u) + sc + 1) & 3) * 8192;
#pragma unroll
                        for (int k = 0; k < 8; ++k) {
                            acc[k >> 1][0] = MFMA(wr[sc][(k & 1) * 2], xf[k], acc[k >> 1][0]);
                            acc[k >> 1][1] = MFMA(wr[sc][(k & 1) * 2 + 1], xf[k], acc[k >> 1][1]);
                            if (sc < 3) xf[k] = *(const LAS u32x4*)(size_t)(an + k * 1024);
                            SB();
                        }
                    }
            }
            done |= ready;
            if (done == 15u) break;
            const bool gu = give_up();
            wait_wr();
            if (gu) break;
            if (wok == 15u) __builtin_amdgcn_s_sleep(1);
            else if (want) {
#pragma unroll
                for (int sc = 0; sc < 4; ++sc) wm[sc] = wmin(sc);
            }
        }
    }
    unsigned gate;
    {
        u32x4 d[4];
#pragma unroll
        for (int q = 0; q < 4; ++q)
#pragma unroll
            for (int k = 0; k < 4; ++k) d[q][k] = nz(pack2(acc[q][k >> 1][(2 * k) & 3], acc[q][k >> 1][(2 * k + 1) & 3]));
        if (bad) {
#pragma unroll
            for (int q = 0; q < 4; ++q) d[q] = u32x4{NAN2, NAN2, NAN2, NAN2};
        }
        auto push = [&](int q) {
            auto rs = RSRC((q == 0 ? dst0 : q == 1 ? dst1 : q == 2 ? dst2 : dst3) + bufoff, isA ? ABUF : CBUF - PBO);
            __builtin_amdgcn_raw_buffer_store_b128(d[q], rs, toff, 0, AUX_SYS);
        };
        if (!early) {
            wait_until(t0, tres);
            reserve(__builtin_amdgcn_s_memrealtime() + R_LATE);
        }
        asm volatile("" : "+v"(slot));
        gate = __builtin_amdgcn_readfirstlane((unsigned)(slot >> 8)) - t0;
        for (;;) {
            const unsigned tn = now32() - t0;
            if (gate - tn - 1u >= 4000u) { gate = tn; break; }
            __builtin_amdgcn_s_sleep(1);
        }
        push(1); push(2); push(3);
        if (!isA) push(0);
    }
    const unsigned gst = isA ? 3 * ABUF : 3 * CBUF;
    auto rg = RSRC(me + (isA ? AOFF : COFF + PBO) + bufoff, isA ? 10u * ABUF : 10u * CBUF - PBO);
    const unsigned so0 = isA ? gst : 0u;
    GAS u32x4* optr = (GAS u32x4*)(outp + ((u * 16 + (lane & 15)) * NDIM + c * 64 + w2 * 32 + g4 * 8) * 2);
    u32x4 in[4];
    wait_until(t0, max(gate + G_MIN, min(gate + G_MAX, G_END)));
    for (;;) {
        in[0] = __builtin_amdgcn_raw_buffer_load_b128(rg, toff, so0, AUX_SYS);
        in[1] = __builtin_amdgcn_raw_buffer_load_b128(rg, toff, gst, AUX_SYS);
        in[2] = __builtin_amdgcn_raw_buffer_load_b128(rg, toff, 2 * gst, AUX_SYS);
        in[3] = __builtin_amdgcn_raw_buffer_load_b128(rg, toff, 3 * gst, AUX_SYS);
        if (__all(min(min(min4(in[0]), min4(in[1])), min(min4(in[2]), min4(in[3]))) != 0)) break;
        if (give_up()) break;
    }
    float y[8];
#pragma unroll
    for (int k = 0; k < 4; ++k) {
        y[2 * k] = isA ? acc[0][k >> 1][(2 * k) & 3] : lo_f(in[0][k]);
        y[2 * k + 1] = isA ? acc[0][k >> 1][(2 * k + 1) & 3] : hi_f(in[0][k]);
    }
#pragma unroll
    for (int q = 1; q < 4; ++q)
#pragma unroll
        for (int k = 0; k < 4; ++k) {
            y[2 * k] += lo_f(in[q][k]);
            y[2 * k + 1] += hi_f(in[q][k]);
        }
    u32x4 o = {pack2(relu(y[0]), relu(y[1])), pack2(relu(y[2]), relu(y[3])),
               pack2(relu(y[4]), relu(y[5])), pack2(relu(y[6]), relu(y[7]))};
    if (bad) o = u32x4{NAN2, NAN2, NAN2, NAN2};
    *optr = o;
}

extern "C" void kernel_launch(int rank, int nranks, ncclComm_t comm,
                              hipStream_t stream,
                              void* const* d_in, const int* in_sizes, int n_in,
                              void* d_out, int out_size,
                              const int* in_dtypes, int out_dtype,
                              void* const* d_out_all,
                              void* const* d_ws_all, size_t ws_bytes) {
    (void)comm; (void)in_dtypes; (void)out_dtype; (void)d_out_all;
    if (nranks != NR || (unsigned)rank >= (unsigned)NR || n_in < 2 || ws_bytes < WS_DATA ||
        in_sizes[0] != MDIM * KLOC || in_sizes[1] != KLOC * NDIM || out_size != MLOC * NDIM) {
        fprintf(stderr, "kernel_launch: unexpected problem shape\n");
        return;
    }
    Params p = {};
    const int g4 = rank & 4, gi = rank & 3;
    const char* x = (const char*)d_in[0];
    const size_t own = (size_t)MLOC * KLOC * 2;
    p.w = (const char*)d_in[1];
    p.out = (char*)d_out;
    p.me = (char*)d_ws_all[rank];
    p.xg[0] = x + g4 * own;
    p.xg[1] = x + (g4 ^ 4) * own;
    for (int q = 0; q < 4; ++q) {
        const int o = (gi + q) & 3;
        p.xl[0][q] = x + (g4 + o) * own;
        p.xl[1][q] = x + ((g4 ^ 4) + o) * own;
        p.dst[0][q] = (char*)d_ws_all[g4 + o] + AOFF + (size_t)(q ? 4 - q : 1) * 3 * ABUF;
        p.dst[1][q] = (char*)d_ws_all[(g4 ^ 4) + o] + COFF + (size_t)gi * 3 * CBUF + PBO;
        p.crs[q] = (char*)d_ws_all[(g4 ^ 4) + q] + COFF + (size_t)gi * 3 * CBUF;
    }
    p.tch[0] = p.me + COFF + 6 * CBUF; p.tch[1] = p.me + COFF + 12 * CBUF - 64; p.tch[2] = p.me + WS_DATA - 64;
    for (int q = 1; q < 4; ++q) p.tch[2 + q] = p.dst[0][q];
    for (int q = 0; q < 4; ++q) p.tch[6 + q] = p.crs[q];
    p.tch[10] = p.out; p.tch[11] = p.out + (size_t)MLOC * NDIM * 2 - 64;
    const char* order = "XXWWXXWWXWXWXWXWXWXWWWXWXWXWXWXWXWXWWWWWWWWWWWWWW";
    unsigned nx = 0, nw = 0, pos = 0;
    for (const char* s = order; *s; ++s, ++pos) {
        const unsigned t = T_FIRST + ((pos * UNIT16) >> 4);
        if (*s == 'X') {
            const unsigned slot = nx < NA ? nx : nx == NA ? 16 : nx - 1;
            p.tx[slot] = t | ((t + (nx <= NA ? LEAD_FAST : LEAD_LAZY)) << 16);
            ++nx;
        } else {
            p.tw[nw] = t | ((t + (nw < NFAST ? LEAD_FAST : LEAD_LAZY)) << 16);
            ++nw;
        }
    }
    for (unsigned xc = 0; xc < 8; ++xc) {
        unsigned ra = 0, rb = 0;
        for (unsigned idx = 0; idx < 16; ++idx) {
            const unsigned u = (xc >> 1) + 4 * (idx & 3), jj = (idx >> 2) * 2 + (xc & 1);
            ((u < NA || (u == NA && jj < NSPLIT)) ? ra : rb)++;
        }
        p.dt[xc] = TILE16 * NWA * 8 / ra;
        p.dt[8 + xc] = TILE16 * NWB * 8 / rb;
    }
    gemm_rs_kernel<<<256, 512, 0, stream>>>(p);
    HIP_CHECK(hipGetLastError());
}
```

```cpp
#include <hip/hip_runtime.h>
#include <rccl/rccl.h>
#include <cstdio>
#include <cstdint>

#define HIP_CHECK(expr) do { hipError_t _e = (expr); if (_e != hipSuccess) { \
    fprintf(stderr, "HIP error %s at %s:%d\n", hipGetErrorString(_e), __FILE__, __LINE__); \
    return; } } while (0)

typedef __attribute__((ext_vector_type(8))) __bf16 bf16x8;
typedef __attribute__((ext_vector_type(4))) __bf16 bf16x4;
typedef __attribute__((ext_vector_type(2))) __bf16 bf16x2;
typedef __attribute__((ext_vector_type(2))) float f32x2;
typedef __attribute__((ext_vector_type(4))) float f32x4;
typedef __attribute__((ext_vector_type(4))) unsigned u32x4;
typedef __attribute__((ext_vector_type(2))) unsigned u32x2;
#define LAS __attribute__((address_space(3)))
#define GAS __attribute__((address_space(1)))

constexpr int NR = 8;
constexpr int MDIM = 2048, NDIM = 2048, KLOC = 256, MLOC = 256;
constexpr unsigned NA = 9;
constexpr unsigned NEARLY = 6;
constexpr unsigned SCOUT = 5;

constexpr unsigned PACER_OFF = 1024, CNT_OFF = 4096;
constexpr unsigned NB = 16 - NA;
constexpr unsigned XIN = 32u * 8192u;
constexpr unsigned PBO = XIN + 16u * 8192u;
constexpr unsigned CBUF = PBO + NB * 65536u;
constexpr unsigned ABUF = NA * 65536u;
constexpr unsigned COFF = 64u << 10;
constexpr unsigned AOFF = COFF + 12u * CBUF - 3u * ABUF;
constexpr unsigned WS_DATA = AOFF + 12u * ABUF;

constexpr unsigned DEADLINE_TICKS = 100000000u;
constexpr int AUX_SYS = 17;
constexpr int L_W = 0, L_XL = 65536, L_XR = 98304, L_FLAG = 131072, L_TOTAL = 131072 + 64;
constexpr unsigned NAN2 = 0x7fc07fc0u;

constexpr unsigned T_BIAS = 100;
constexpr unsigned T_FIRST = T_BIAS + 60;
constexpr unsigned UNIT16 = 220;
constexpr unsigned PUSH16 = 56, PULL16 = 52;
constexpr unsigned C_GATE = 22;
constexpr unsigned A_GATE = 1;
constexpr unsigned T_ZERO = T_BIAS + 200, D_ZERO = 150;
constexpr unsigned S_LEAD = 250;
constexpr unsigned LEAD_FAST = 40, LEAD_LAZY = 110, NFAST = 10;
constexpr unsigned R_A0 = T_BIAS + 180, R_B0 = T_BIAS + 540;
constexpr unsigned R_LEAD = 15, R_BLOCK = 14, R_LATE = 30;
constexpr unsigned G_MIN = 50, G_MAX = 250, G_END = T_BIAS + 1430;
constexpr unsigned L_A0 = 50, L_AJ = 10, L_B0 = T_BIAS + 130;
constexpr unsigned L_XFIRST = 250, L_W1 = 145, L_W2 = T_BIAS + 380, L_W3 = T_BIAS + 440;

struct Params {
    const char* w;
    char* me;
    const char* xg[2];
    const char* xl[2][4];
    char* dst[2][4];
    char* crs[4];
    char* out;
    unsigned dt[16];
    unsigned park[4];
    unsigned tx[16];
    unsigned tw[32];
    char* tch[12];
};

__device__ __forceinline__ unsigned pack2(float a, float b) {
    f32x2 v = {a, b};
    return __builtin_bit_cast(unsigned, __builtin_convertvector(v, bf16x2));
}
__device__ __forceinline__ unsigned nz(unsigned d) { return d | ((d - 1u) & 0x80000000u); }
__device__ __forceinline__ u32x4 nz4(u32x4 d) { return u32x4{nz(d[0]), nz(d[1]), nz(d[2]), nz(d[3])}; }
__device__ __forceinline__ float lo_f(unsigned d) { return __builtin_bit_cast(float, d << 16); }
__device__ __forceinline__ float hi_f(unsigned d) { return __builtin_bit_cast(float, d & 0xffff0000u); }
__device__ __forceinline__ float relu(float y) { return y < 0.f ? 0.f : y; }
__device__ __forceinline__ unsigned min4(u32x4 v) { return min(min(v[0], v[1]), min(v[2], v[3])); }
__device__ __forceinline__ unsigned now32() { return (unsigned)__builtin_amdgcn_s_memrealtime(); }
__device__ __forceinline__ void wait_until(unsigned t0, unsigned d) {
    while (now32() - t0 < d) __builtin_amdgcn_s_sleep(1);
}
__device__ __forceinline__ u32x4 tr_frag(const char* a, int hi_off) {
    u32x2 lo = __builtin_bit_cast(u32x2, __builtin_amdgcn_ds_read_tr16_b64_v4bf16((LAS bf16x4*)a));
    u32x2 hi = __builtin_bit_cast(u32x2, __builtin_amdgcn_ds_read_tr16_b64_v4bf16((LAS bf16x4*)(a + hi_off)));
    return u32x4{lo[0], lo[1], hi[0], hi[1]};
}
#define MFMA(a, bb, c) __builtin_amdgcn_mfma_f32_16x16x32_bf16(__builtin_bit_cast(bf16x8, a), __builtin_bit_cast(bf16x8, bb), c, 0, 0, 0)
#define WAIT_VM0() __builtin_amdgcn_s_waitcnt(0x0F70)
#define WAIT_LGKM0() __builtin_amdgcn_s_waitcnt(0xC07F)
#define SGB(mask, n) __builtin_amdgcn_sched_group_barrier(mask, n, 0)
#define SB() __builtin_amdgcn_sched_barrier(0)
#define RSRC(base, bytes) __builtin_amdgcn_make_buffer_rsrc((base), 0, (int)(bytes), 0x00020000)
#define GLDS(src, dstlds) __builtin_amdgcn_global_load_lds((const GAS void*)(src), (LAS void*)(dstlds), 16, 0, 0)

__device__ __forceinline__ void flag_add(char* lds, int f, unsigned lane) {
    const unsigned a = (unsigned)(size_t)(LAS char*)lds + L_FLAG + f * 4, one = 1;
    if (lane == 0) asm volatile("ds_add_u32 %0, %1" :: "v"(a), "v"(one) : "memory");
}
__device__ __forceinline__ void flag_wait(char* lds, int f, unsigned n) {
    const unsigned a = (unsigned)(size_t)(LAS char*)lds + L_FLAG + f * 4;
    for (;;) {
        unsigned v;
        asm volatile("ds_read_b32 %0, %1\n\ts_waitcnt lgkmcnt(0)" : "=&v"(v) : "v"(a) : "memory");
        if ((unsigned)__builtin_amdgcn_readfirstlane(v) >= n) break;
        __builtin_amdgcn_s_sleep(1);
    }
}
enum { F_XR = 0  , F_LA = 4, F_LB = 5, F_WL = 6  , F_FREE = 10   };

__global__ __launch_bounds__(512, 2) void gemm_rs_kernel(Params p) {
    __shared__ __attribute__((aligned(16))) char lds[L_TOTAL];
    unsigned long long ta, tb;
    asm volatile("s_memrealtime %0" : "=s"(ta));
    SB();
    const char* wg = p.w; char* me = p.me; const char* xg0 = p.xg[0]; const char* xg1 = p.xg[1];
#define TIME_BASE() \
    const unsigned t0 = (unsigned)tb + min((unsigned)(tb - ta) >> 1, T_BIAS) - T_BIAS
    const unsigned lane = threadIdx.x & 63;
    const unsigned wave = __builtin_amdgcn_readfirstlane(threadIdx.x >> 6);
    const unsigned b = blockIdx.x;
    const unsigned xcd = b & 7, idx = b >> 4;
    const unsigned g4 = lane >> 4, q4 = (lane >> 2) & 3, pc = lane & 3;

    if (__builtin_expect((b & 8) == 0, 1)) {
        const bool isW = wave < 2;
        const unsigned c = 2 * idx + (wave & 1), ss = xcd;
        const unsigned peer = xcd >> 1, xo = (xcd & 1) * 2 + (wave & 1);
        const bool xa = idx < NA;
        const unsigned mw = 0u - (unsigned)isW;
        const unsigned soff = (mw & ((32 * ss * NDIM + 64 * c) * 2)) | (~mw & (((xo * MLOC + 16 * idx) * KLOC + peer * 64) * 2));
        const unsigned voff = (mw & ((lane >> 3) * (NDIM * 2) + (lane & 7) * 16)) | (~mw & ((lane & 15) * (KLOC * 2) + g4 * 16));
        const unsigned st1 = (mw & (8 * NDIM * 2)) | (~mw & 64), st2 = mw & (16 * NDIM * 2);
        const char* sbase = (isW ? wg : xa ? xg1 : xg0) + soff;
        asm volatile("s_memrealtime %0" : "=s"(tb), "+s"(sbase));
        u32x4 v[4];
        auto load = [&]() {
            v[0] = *(const GAS u32x4*)(sbase + voff);
            v[1] = *(const GAS u32x4*)(sbase + st1 + voff);
            v[2] = *(const GAS u32x4*)(sbase + st2 + voff);
            v[3] = *(const GAS u32x4*)(sbase + st1 + st2 + voff);
        };
        const bool fast = idx < NEARLY && wave < 4;
        if (fast) load();
        asm volatile("s_waitcnt lgkmcnt(0)" : "+s"(ta), "+s"(tb) :: "memory");
        TIME_BASE();
        char* pws = p.crs[peer];
        if (wave == 4 && idx == SCOUT) {
            unsigned sink = __hip_atomic_load((const GAS unsigned*)me, __ATOMIC_RELAXED, __HIP_MEMORY_SCOPE_AGENT);
            sink += __hip_atomic_load((const GAS unsigned*)(pws + (lane & 1) * (3 * CBUF - 64)), __ATOMIC_RELAXED, __HIP_MEMORY_SCOPE_SYSTEM);
            asm volatile("" :: "v"(sink));
#pragma unroll 1
            for (int i = 0; i < 12; ++i) {
                const unsigned len = i < 3 ? 0u : i < 6 ? 3 * ABUF - 64 : i < 10 ? 3 * CBUF - 64 : 0u;
                const unsigned t_ = __hip_atomic_load((const GAS unsigned*)(p.tch[i] + (lane & 1) * len), __ATOMIC_RELAXED, __HIP_MEMORY_SCOPE_SYSTEM);
                asm volatile("" :: "v"(t_));
            }
        }
        GAS unsigned* cnt = (GAS unsigned*)(me + CNT_OFF) + (b * 8 + wave);
        const unsigned cval = __hip_atomic_load(cnt, __ATOMIC_RELAXED, __HIP_MEMORY_SCOPE_AGENT);
        const unsigned tsend = (isW ? p.tw[c] : p.tx[idx]) & 0xffffu;
        if (wave < 4) {
            if (!fast) {
                wait_until(t0, max(tsend, S_LEAD) - S_LEAD);
                load();
            }
            const unsigned when = tsend + (((isW ? (ss & 1) * 2 : xo) * (UNIT16 / 4)) >> 4);
            if (isW) {
                char* sl = lds + wave * 4096;
#pragma unroll
                for (int q = 0; q < 4; ++q) *(u32x4*)(sl + q * 1024 + lane * 16) = v[q];
                const char* base = sl + (8 * g4 + q4) * 128 + pc * 16;
                v[0] = tr_frag(base, 512); v[1] = tr_frag(base + 8, 512);
                v[2] = tr_frag(base + 64, 512); v[3] = tr_frag(base + 72, 512);
            }
            const unsigned dst = (isW ? c * 8192u + (ss & 1) * 2048u : XIN + (idx * 4 + ((xo - peer) & 3)) * 2048u) + lane * 16u;
#pragma unroll
            for (int q = 0; q < 4; ++q) v[q] = nz4(v[q]);
            auto rp = RSRC(pws + ((__builtin_amdgcn_readfirstlane(cval) + 1u) % 3u) * CBUF, PBO);
            wait_until(t0, when);
            __builtin_amdgcn_raw_buffer_store_b128(v[0], rp, dst, 0, AUX_SYS);
            __builtin_amdgcn_raw_buffer_store_b128(v[1], rp, dst + 1024, 0, AUX_SYS);
            if (isW) {
                __builtin_amdgcn_raw_buffer_store_b128(v[2], rp, dst + 4096, 0, AUX_SYS);
                __builtin_amdgcn_raw_buffer_store_b128(v[3], rp, dst + 4096 + 1024, 0, AUX_SYS);
            }
        }
        unsigned cv2 = cval;
        asm volatile("" : "+v"(cv2));
        const unsigned call = __builtin_amdgcn_readfirstlane(cv2) + 1;
        if (lane == 0) __hip_atomic_store(cnt, call, __ATOMIC_RELAXED, __HIP_MEMORY_SCOPE_AGENT);
        if (wave >= 4) {
            auto rz = RSRC(me, WS_DATA);
            const u32x4 z = {0u, 0u, 0u, 0u};
            const unsigned zb = (call + 2) % 3, zid = (idx * 8 + xcd) * 4 + (wave - 4);
            const unsigned zph = ((idx * 4 + (wave - 4)) * D_ZERO) >> 6;
            constexpr unsigned NCP = PBO / 1024, NAP = ABUF / 1024;
            static_assert(4 * NCP + 3 * NAP <= 7 * 512, "re-arm loop too short");
#pragma unroll 1
            for (unsigned q = 0; q < 7; ++q) {
                const unsigned id = q * 512 + zid;
                if (id >= 4 * NCP + 3 * NAP) break;
                const unsigned i2 = id - 4 * NCP;
                const unsigned off = id < 4 * NCP ? COFF + ((id / NCP) * 3 + zb) * CBUF + (id % NCP) * 1024u
                                                  : AOFF + ((i2 / NAP + 1) * 3 + zb) * ABUF + (i2 % NAP) * 1024u;
                wait_until(t0, T_ZERO + zph + q * D_ZERO);
                __builtin_amdgcn_raw_buffer_store_b128(z, rz, off + lane * 16u, 0, AUX_SYS);
            }
        }
        return;
    }

    const unsigned u = (xcd >> 1) + 4 * (idx & 3);
    const unsigned jj = (idx >> 2) * 2 + (xcd & 1);
    const unsigned grp = wave >> 1, w2 = wave & 1;
    const unsigned c = jj + 8 * grp;
    const bool isA = u < NA;
    const unsigned ci = isA ? 0 : 1;
    const unsigned wlane = (lane >> 3) * (NDIM * 2) + ((lane & 7) ^ (((lane >> 4) & 1) << 2)) * 16;
    const char* xl0 = p.xl[ci][0]; const char* xl1 = p.xl[ci][1]; const char* xl2 = p.xl[ci][2]; const char* xl3 = p.xl[ci][3];
    {
        const unsigned zero = 0, fa = (unsigned)(size_t)(LAS char*)lds + L_FLAG + (lane & 15) * 4;
        asm volatile("ds_write_b32 %8, %9\n\ts_memrealtime %7\n\ts_waitcnt lgkmcnt(0)\n\ts_barrier"
                     : "+s"(wg), "+s"(me), "+s"(xl0), "+s"(xl1), "+s"(xl2), "+s"(xl3), "+s"(ta), "=&s"(tb)
                     : "v"(fa), "v"(zero) : "memory");
    }
    TIME_BASE();
    const unsigned tp0 = isA ? L_A0 + jj * L_AJ : L_B0;
    if (grp == 1) {
        if (tp0 > L_A0) wait_until(t0, tp0);
        const unsigned xlane = (16 * u + (lane >> 3)) * (KLOC * 2) + ((lane & 7) ^ (lane >> 3)) * 16 + w2 * 128;
        const char* wsrc = wg + (64 * w2 * NDIM + 64 * jj) * 2;
#pragma unroll
        for (int h = 0; h < 2; ++h) {
#pragma unroll
            for (int i = 0; i < 8; ++i)
                GLDS(wsrc + (size_t)(h * 128 + i * 8) * NDIM * 2 + wlane, lds + L_W + h * 16384 + w2 * 8192 + i * 1024);
#pragma unroll
            for (int q = 0; q < 4; ++q) {
                const char* xs = (q == 0 ? xl0 : q == 1 ? xl1 : q == 2 ? xl2 : xl3) + h * 256;
                GLDS(xs + xlane, lds + L_XL + q * 8192 + h * 4096 + w2 * 2048);
                GLDS(xs + 8 * KLOC * 2 + xlane, lds + L_XL + q * 8192 + h * 4096 + w2 * 2048 + 1024);
            }
        }
    }
    GAS unsigned* cnt = (GAS unsigned*)(me + CNT_OFF) + (b * 8 + wave);
    GAS unsigned* tmo = (GAS unsigned*)me;
    const unsigned cval = __hip_atomic_load(cnt, __ATOMIC_RELAXED, __HIP_MEMORY_SCOPE_AGENT);
    const unsigned tval = __hip_atomic_load(tmo, __ATOMIC_RELAXED, __HIP_MEMORY_SCOPE_AGENT);
    char* dst0 = p.dst[ci][0]; char* dst1 = p.dst[ci][1]; char* dst2 = p.dst[ci][2]; char* dst3 = p.dst[ci][3];
    char* outp = p.out;
    unsigned dtx = p.dt[ci * 8 + xcd], txu = p.tx[u], twc = p.tw[c];
    unsigned pk0 = p.park[0], pk1 = p.park[1], pk2 = p.park[2], pk3 = p.park[3];
    asm volatile("" : "+s"(dst0), "+s"(dst1), "+s"(dst2), "+s"(dst3), "+s"(outp), "+s"(dtx), "+s"(txu), "+s"(twc),
                      "+s"(pk0), "+s"(pk1), "+s"(pk2), "+s"(pk3));
    txu >>= 16;
    twc >>= 16;
    if (grp == 1) {
        asm volatile("s_waitcnt vmcnt(18)" ::: "memory");
        flag_add(lds, F_LA, lane);
    }
    const char* xl_rd0 = lds + L_XL + (lane & 15) * 128 + (g4 ^ (lane & 7)) * 16;
    const char* xl_rd1 = lds + L_XL + (lane & 15) * 128 + ((4 + g4) ^ (lane & 7)) * 16;
    const char* w_rd = lds + L_W + (grp & 1) * 32768 + (8 * g4 + q4) * 128 + ((w2 ^ (q4 >> 1)) * 64) + pc * 16;
    const unsigned wroff = (c * 2 + w2) * 4096u + lane * 16u;
    const unsigned toff = (((isA ? (c >> 1) * NA + u : (c >> 1) * NB + (u - NA)) * 2 + (c & 1)) * 2 + w2) * 1024u + lane * 16u;
    u32x4 wr[4][4];
#pragma unroll
    for (int k = 0; k < 16; ++k) wr[k >> 2][k & 3] = u32x4{0u, 0u, 0u, 0u};
    WAIT_VM0();
    if (grp == 1) flag_add(lds, F_LB, lane);
    const unsigned call = __builtin_amdgcn_readfirstlane(cval) + 1;
    const unsigned bufi = call % 3;
    const unsigned bufoff = bufi * (isA ? ABUF : CBUF);
    bool bad = __builtin_amdgcn_readfirstlane(tval) != 0;
    if (lane == 0) __hip_atomic_store(cnt, call, __ATOMIC_RELAXED, __HIP_MEMORY_SCOPE_AGENT);
    char* cin = me + COFF + bufi * CBUF;
    auto rl = RSRC(cin, 10u * CBUF);
    if (!isA) {
        auto rzb = RSRC(me + COFF + PBO + ((call + 1) % 3) * CBUF, 10u * CBUF - PBO);
        const u32x4 z = {0u, 0u, 0u, 0u};
#pragma unroll
        for (int q = 0; q < 4; ++q) __builtin_amdgcn_raw_buffer_store_b128(z, rzb, toff + q * (3 * CBUF), 0, AUX_SYS);
    }
    auto give_up = [&]() -> bool {
        const unsigned el = now32() - t0;
        if (!bad && el < 10000u) return false;
        if (bad || el > DEADLINE_TICKS ||
            __builtin_amdgcn_readfirstlane(__hip_atomic_load(tmo, __ATOMIC_RELAXED, __HIP_MEMORY_SCOPE_AGENT)) != 0) {
            if (lane == 0) __hip_atomic_store(tmo, call, __ATOMIC_RELAXED, __HIP_MEMORY_SCOPE_AGENT);
            bad = true;
            return true;
        }
        __builtin_amdgcn_s_sleep(64);
        return false;
    };
    if (grp >= 2) {
        const unsigned src = wave - 4;
        auto load_w1 = [&]() {
            wait_until(t0, tp0 + L_W1);
            const char* wsrc = wg + (64 * src * NDIM + 64 * (jj + 8)) * 2;
#pragma unroll
            for (int i = 0; i < 8; ++i) GLDS(wsrc + (size_t)i * 8 * NDIM * 2 + wlane, lds + L_W + 32768 + src * 8192 + i * 1024);
            WAIT_VM0();
            flag_add(lds, F_WL + 1, lane);
        };
        const bool xfirst = txu <= tp0 + L_XFIRST;
        if (!xfirst) load_w1();
        wait_until(t0, txu);
        const unsigned xoff = src * (3 * CBUF) + XIN + u * 8192u;
        u32x4 xr[8];
        for (;;) {
            unsigned mn = 0xffffffffu;
#pragma unroll
            for (int i = 0; i < 8; ++i) xr[i] = __builtin_amdgcn_raw_buffer_load_b128(rl, xoff + i * 1024u + lane * 16u, 0, AUX_SYS);
#pragma unroll
            for (int i = 0; i < 8; ++i) mn = min(mn, min4(xr[i]));
            if (__all(mn != 0)) break;
            if (give_up()) {
#pragma unroll
                for (int i = 0; i < 8; ++i) xr[i] = u32x4{NAN2, NAN2, NAN2, NAN2};
                break;
            }
            for (;;) {
                __builtin_amdgcn_s_sleep(2);
                const unsigned cv = __builtin_amdgcn_raw_buffer_load_b32(rl, xoff + (lane & 7) * 1024u + 1020u, 0, AUX_SYS);
                if (__all(cv != 0) || give_up()) break;
            }
        }
#pragma unroll
        for (int i = 0; i < 8; ++i) *(u32x4*)(lds + L_XR + src * 8192 + i * 1024 + lane * 16) = xr[i];
        WAIT_LGKM0();
        flag_add(lds, F_XR + src, lane);
        if (xfirst) load_w1();
        flag_wait(lds, F_FREE + (grp & 1), 2);
        wait_until(t0, grp == 2 ? L_W2 : L_W3);
        const char* wsrc = wg + (128 * w2 * NDIM + 64 * c) * 2;
#pragma unroll
        for (int i = 0; i < 16; ++i) GLDS(wsrc + (size_t)i * 8 * NDIM * 2 + wlane, lds + L_W + (grp & 1) * 32768 + w2 * 16384 + i * 1024);
        WAIT_VM0();
        flag_add(lds, F_WL + grp, lane);
    }
    const u32x4 rlw = {(unsigned)(size_t)cin, (unsigned)((size_t)cin >> 32) & 0xffffu, 10u * CBUF, 0x00020000u};
#define WR_ALL "+v"(wr[0][0]), "+v"(wr[0][1]), "+v"(wr[0][2]), "+v"(wr[0][3]), "+v"(wr[1][0]), "+v"(wr[1][1]), "+v"(wr[1][2]), "+v"(wr[1][3]), \
               "+v"(wr[2][0]), "+v"(wr[2][1]), "+v"(wr[2][2]), "+v"(wr[2][3]), "+v"(wr[3][0]), "+v"(wr[3][1]), "+v"(wr[3][2]), "+v"(wr[3][3])
    auto load_wr = [&](unsigned mask) {
        const unsigned s1 = 3 * CBUF, s2 = 6 * CBUF, s3 = 9 * CBUF;
#define LD4(a, b_, c_, d_, so) \
        "buffer_load_dwordx4 %" #a ", %16, %17, " so " offen sc0 sc1\n\t" \
        "buffer_load_dwordx4 %" #b_ ", %16, %17, " so " offen offset:1024 sc0 sc1\n\t" \
        "buffer_load_dwordx4 %" #c_ ", %16, %17, " so " offen offset:2048 sc0 sc1\n\t" \
        "buffer_load_dwordx4 %" #d_ ", %16, %17, " so " offen offset:3072 sc0 sc1\n\t"
        asm volatile("s_nop 4\n\t"
                     "s_bitcmp0_b32 %18, 0\n\ts_cbranch_scc1 1f\n\t" LD4(0, 1, 2, 3, "0")
                     "1:\n\ts_bitcmp0_b32 %18, 1\n\ts_cbranch_scc1 2f\n\t" LD4(4, 5, 6, 7, "%19")
                     "2:\n\ts_bitcmp0_b32 %18, 2\n\ts_cbranch_scc1 3f\n\t" LD4(8, 9, 10, 11, "%20")
                     "3:\n\ts_bitcmp0_b32 %18, 3\n\ts_cbranch_scc1 4f\n\t" LD4(12, 13, 14, 15, "%21")
                     "4:"
                     : WR_ALL : "v"(wroff), "s"(rlw), "s"(mask), "s"(s1), "s"(s2), "s"(s3) : "memory", "scc");
    };
    auto wait_wr = [&]() { asm volatile("s_waitcnt vmcnt(0)" : WR_ALL :: "memory"); };
    auto wmin = [&](int sc) { return min(min(min4(wr[sc][0]), min4(wr[sc][1])), min(min4(wr[sc][2]), min4(wr[sc][3]))); };
    const bool pref = twc <= now32() - t0 + 25u;
    if (pref) wait_until(t0, twc);
    load_wr(__builtin_amdgcn_readfirstlane(pref ? 15u : 0u));
    unsigned tres = isA ? R_A0 + ((((c & ~1u) * NA + 2u * u) * PUSH16) >> 4)
                        : R_B0 + ((((c & ~1u) * NB + 2u * (u - NA)) * PULL16) >> 4);
    unsigned xr_a = (unsigned)(size_t)(LAS char*)lds + L_XR + lane * 16;
    unsigned fa = (unsigned)(size_t)(LAS char*)lds + L_FLAG + F_XR * 4 + (lane & 3) * 4;
    char* pacer0 = me + PACER_OFF + (ci * 8 + xcd) * 128;
    asm volatile("" : "+s"(tres), "+s"(pacer0), "+v"(xr_a), "+v"(fa));
    if (grp >= 1) { flag_wait(lds, F_WL + grp, grp == 1 ? 4 : 2); flag_wait(lds, F_LB, 2); }
    flag_wait(lds, F_LA, 2);

    f32x4 acc[4][2];
    unsigned wm[4];
    {
#pragma unroll
        for (int q = 0; q < 4; ++q) { acc[q][0] = f32x4{0.f, 0.f, 0.f, 0.f}; acc[q][1] = acc[q][0]; }
        u32x4 wf[2][2], xf[2][4];
        auto rdl = [&](int s) {
            wf[s & 1][0] = tr_frag(w_rd + s * 4096, 512);
            xf[s & 1][0] = *(const u32x4*)(((s & 1) ? xl_rd1 : xl_rd0) + (0 * 4 + (s >> 1)) * 2048);
            wf[s & 1][1] = tr_frag(w_rd + s * 4096 + 8, 512);
#pragma unroll
            for (int q = 1; q < 4; ++q)
                xf[s & 1][q] = *(const u32x4*)(((s & 1) ? xl_rd1 : xl_rd0) + (q * 4 + (s >> 1)) * 2048);
        };
        rdl(0);
        SB();
#pragma unroll
        for (int s = 0; s < 8; ++s) {
            if (s == 3) { flag_wait(lds, F_LB, 2); SB(); }
            if (s == 6) { wait_wr(); SB(); }
            if (s + 1 < 8) rdl(s + 1);
#pragma unroll
            for (int q = 0; q < 4; ++q) {
                acc[q][0] = MFMA(wf[s & 1][0], xf[s & 1][q], acc[q][0]);
                acc[q][1] = MFMA(wf[s & 1][1], xf[s & 1][q], acc[q][1]);
            }
            if (s >= 6) { wm[2 * (s - 6)] = wmin(2 * (s - 6)); wm[2 * (s - 6) + 1] = wmin(2 * (s - 6) + 1); }
            if (s < 6) {
                SGB(0x100, 2); SGB(0x8, 1); SGB(0x100, 2); SGB(0x8, 1); SGB(0x100, 1); SGB(0x8, 1); SGB(0x100, 1); SGB(0x8, 1);
                SGB(0x100, 1); SGB(0x8, 1); SGB(0x100, 1); SGB(0x8, 3);
            } else if (s == 6) {
                SGB(0x100, 2); SGB(0x8, 1); SGB(0x2, 2); SGB(0x100, 2); SGB(0x8, 1); SGB(0x2, 2); SGB(0x100, 1); SGB(0x8, 1); SGB(0x2, 2);
                SGB(0x100, 1); SGB(0x8, 1); SGB(0x2, 2); SGB(0x100, 1); SGB(0x8, 1); SGB(0x2, 2); SGB(0x100, 1); SGB(0x8, 1); SGB(0x2, 2);
                SGB(0x8, 1); SGB(0x2, 2); SGB(0x8, 1); SGB(0x2, 2);
            } else {
                SGB(0x8, 1); SGB(0x2, 2); SGB(0x8, 1); SGB(0x2, 2); SGB(0x8, 1); SGB(0x2, 2); SGB(0x8, 1); SGB(0x2, 2);
                SGB(0x8, 1); SGB(0x2, 2); SGB(0x8, 1); SGB(0x2, 2); SGB(0x8, 1); SGB(0x2, 2); SGB(0x8, 1); SGB(0x2, 2);
            }
            SB();
            if (s >= 6) asm volatile("" : "+v"(wm[2 * (s - 6)]), "+v"(wm[2 * (s - 6) + 1]));
        }
        if (grp < 2) flag_add(lds, F_FREE + grp, lane);
    }
    unsigned long long slot = 0;
    auto reserve = [&](unsigned long long want) {
        unsigned vz = 0;
        asm volatile("" : "+v"(vz));
        GAS unsigned long long* pacer = (GAS unsigned long long*)(pacer0 + vz);
        if (lane == 0) {
            __hip_atomic_fetch_max(pacer, want << 8, __ATOMIC_RELAXED, __HIP_MEMORY_SCOPE_WORKGROUP);
            slot = __hip_atomic_fetch_add(pacer, (unsigned long long)dtx, __ATOMIC_RELAXED, __HIP_MEMORY_SCOPE_WORKGROUP);
        }
    };
    bool early = false;
    {
        unsigned done = 0;
        bool tried = pref;
        for (;;) {
            unsigned fl;
            asm volatile("ds_read_b32 %0, %1" : "=v"(fl) : "v"(fa) : "memory");
            unsigned wok = 0;
#pragma unroll
            for (int sc = 0; sc < 4; ++sc) if (__all(wm[sc] != 0)) wok |= 1u << sc;
            wok = __builtin_amdgcn_readfirstlane(wok);
            if (wok == 0) wait_until(t0, twc);
            unsigned want = ~wok & 15u;
            if (want && tried) {
                const unsigned cv = __builtin_amdgcn_raw_buffer_load_b32(rl, q4 * (3 * CBUF) + (c * 2 + w2) * 4096u + pc * 1024u + 1020u, 0, AUX_SYS);
                const unsigned zb_ = (unsigned)__builtin_amdgcn_ballot_w64(cv == 0);
                want &= ((zb_ & 0xfu) ? 0u : 1u) | ((zb_ & 0xf0u) ? 0u : 2u) | ((zb_ & 0xf00u) ? 0u : 4u) | ((zb_ & 0xf000u) ? 0u : 8u);
                if (!want) __builtin_amdgcn_s_sleep(2);
            }
            tried = true;
            load_wr(want);
            asm volatile("s_waitcnt lgkmcnt(0)" : "+v"(fl) :: "memory");
            const unsigned ready = wok & (unsigned)__builtin_amdgcn_ballot_w64(fl != 0) & ~done & 15u;
            if (ready) {
                u32x4 xf[8];
                const unsigned a0 = xr_a + __builtin_ctz(ready) * 8192;
#pragma unroll
                for (int k = 0; k < 8; ++k) xf[k] = *(const LAS u32x4*)(size_t)(a0 + k * 1024);
                SB();
                if ((ready | done) == 15u) {
                    const unsigned long long tn = __builtin_amdgcn_s_memrealtime();
                    const unsigned ahead = R_LEAD + R_BLOCK * __builtin_popcount(ready);
                    early = isA && (unsigned)tn - t0 + ahead >= tres;
                    if (early) reserve(tn + ahead);
                }
#pragma unroll
                for (int sc = 0; sc < 4; ++sc)
                    if ((ready >> sc) & 1) {
                        const unsigned an = xr_a + ((__builtin_ctz((ready >> (sc + 1)) | 8u) + sc + 1) & 3) * 8192;
#pragma unroll
                        for (int k = 0; k < 8; ++k) {
                            acc[k >> 1][0] = MFMA(wr[sc][(k & 1) * 2], xf[k], acc[k >> 1][0]);
                            acc[k >> 1][1] = MFMA(wr[sc][(k & 1) * 2 + 1], xf[k], acc[k >> 1][1]);
                            if (sc < 3) xf[k] = *(const LAS u32x4*)(size_t)(an + k * 1024);
                            SB();
                        }
                    }
            }
            done |= ready;
            if (done == 15u) break;
            const bool gu = give_up();
            wait_wr();
            if (gu) break;
            if (wok == 15u) __builtin_amdgcn_s_sleep(1);
            else if (want) {
#pragma unroll
                for (int sc = 0; sc < 4; ++sc) wm[sc] = wmin(sc);
            }
        }
    }
    auto wait_slot = [&]() -> unsigned {
        asm volatile("" : "+v"(slot));
        const unsigned g = __builtin_amdgcn_readfirstlane((unsigned)(slot >> 8)) - t0;
        for (;;) {
            const unsigned tn = now32() - t0;
            if (g - tn - 1u >= 4000u) return tn;
            __builtin_amdgcn_s_sleep(1);
        }
    };
    {
        u32x4 d[4];
#pragma unroll
        for (int q = 0; q < 4; ++q)
#pragma unroll
            for (int k = 0; k < 4; ++k) d[q][k] = nz(pack2(acc[q][k >> 1][(2 * k) & 3], acc[q][k >> 1][(2 * k + 1) & 3]));
        if (bad) {
#pragma unroll
            for (int q = 0; q < 4; ++q) d[q] = u32x4{NAN2, NAN2, NAN2, NAN2};
        }
        if (!isA) {
            auto rs = RSRC(me + COFF + PBO + bufoff, 10u * CBUF - PBO);
            __builtin_amdgcn_raw_buffer_store_b128(d[0], rs, toff + pk0, 0, AUX_SYS);
            __builtin_amdgcn_raw_buffer_store_b128(d[1], rs, toff + pk1, 0, AUX_SYS);
            __builtin_amdgcn_raw_buffer_store_b128(d[2], rs, toff + pk2, 0, AUX_SYS);
            __builtin_amdgcn_raw_buffer_store_b128(d[3], rs, toff + pk3, 0, AUX_SYS);
        }
        if (!early) {
            wait_until(t0, tres);
            if (!isA) {
                const unsigned ca = A_GATE + (c >> 1), ql = max(lane & 3u, 1u);
                const unsigned go = (lane & 4) ? AOFF + (ql * 3 + bufi) * ABUF + ((ca >> 1) * NA * 2 + (ca & 1)) * 2048u + 2044u
                                               : COFF + (pc * 3 + bufi) * CBUF + C_GATE * 8192u + 8188u;
                auto rw = RSRC(me, WS_DATA);
                for (;;) {
                    const unsigned cv = __builtin_amdgcn_raw_buffer_load_b32(rw, go, 0, AUX_SYS);
                    if (__all(cv != 0) || give_up()) break;
                    __builtin_amdgcn_s_sleep(4);
                }
            }
            reserve(__builtin_amdgcn_s_memrealtime() + R_LATE);
        }
        const unsigned gate = wait_slot();
        if (isA) {
            __builtin_amdgcn_raw_buffer_store_b128(d[1], RSRC(dst1 + bufoff, ABUF), toff, 0, AUX_SYS);
            __builtin_amdgcn_raw_buffer_store_b128(d[2], RSRC(dst2 + bufoff, ABUF), toff, 0, AUX_SYS);
            __builtin_amdgcn_raw_buffer_store_b128(d[3], RSRC(dst3 + bufoff, ABUF), toff, 0, AUX_SYS);
            wait_until(t0, max(gate + G_MIN, min(gate + G_MAX, G_END)));
        }
    }
    char* ab = me + AOFF;
    const unsigned gbytes = isA ? ABUF : CBUF - PBO;
    auto rg0 = RSRC((isA ? ab + 3 * ABUF : dst0) + bufoff, gbytes), rg1 = RSRC((isA ? ab + 3 * ABUF : dst1) + bufoff, gbytes);
    auto rg2 = RSRC((isA ? ab + 6 * ABUF : dst2) + bufoff, gbytes), rg3 = RSRC((isA ? ab + 9 * ABUF : dst3) + bufoff, gbytes);
    GAS u32x4* optr = (GAS u32x4*)(outp + ((u * 16 + (lane & 15)) * NDIM + c * 64 + w2 * 32 + g4 * 8) * 2);
    u32x4 in[4];
    for (;;) {
        in[0] = __builtin_amdgcn_raw_buffer_load_b128(rg0, toff, 0, AUX_SYS);
        in[1] = __builtin_amdgcn_raw_buffer_load_b128(rg1, toff, 0, AUX_SYS);
        in[2] = __builtin_amdgcn_raw_buffer_load_b128(rg2, toff, 0, AUX_SYS);
        in[3] = __builtin_amdgcn_raw_buffer_load_b128(rg3, toff, 0, AUX_SYS);
        if (__all(min(min(min4(in[0]), min4(in[1])), min(min4(in[2]), min4(in[3]))) != 0)) break;
        if (give_up()) break;
        if (!isA) {
            const GAS unsigned* cp = (const GAS unsigned*)((pc == 0 ? dst0 : pc == 1 ? dst1 : pc == 2 ? dst2 : dst3) + bufoff + (toff & ~1023u) + 1020u);
            for (;;) {
                __builtin_amdgcn_s_sleep(32);
                unsigned cv = 1;
                if (lane < 4) cv = __hip_atomic_load(cp, __ATOMIC_RELAXED, __HIP_MEMORY_SCOPE_SYSTEM);
                if (__all(cv != 0) || give_up()) break;
            }
            reserve(__builtin_amdgcn_s_memrealtime() + R_LATE);
            wait_slot();
        }
    }
    float y[8];
#pragma unroll
    for (int k = 0; k < 4; ++k) {
        y[2 * k] = isA ? acc[0][k >> 1][(2 * k) & 3] : lo_f(in[0][k]);
        y[2 * k + 1] = isA ? acc[0][k >> 1][(2 * k + 1) & 3] : hi_f(in[0][k]);
    }
#pragma unroll
    for (int q = 1; q < 4; ++q)
#pragma unroll
        for (int k = 0; k < 4; ++k) {
            y[2 * k] += lo_f(in[q][k]);
            y[2 * k + 1] += hi_f(in[q][k]);
        }
    u32x4 o = {pack2(relu(y[0]), relu(y[1])), pack2(relu(y[2]), relu(y[3])),
               pack2(relu(y[4]), relu(y[5])), pack2(relu(y[6]), relu(y[7]))};
    if (bad) o = u32x4{NAN2, NAN2, NAN2, NAN2};
    *optr = o;
}

extern "C" void kernel_launch(int rank, int nranks, ncclComm_t comm,
                              hipStream_t stream,
                              void* const* d_in, const int* in_sizes, int n_in,
                              void* d_out, int out_size,
                              const int* in_dtypes, int out_dtype,
                              void* const* d_out_all,
                              void* const* d_ws_all, size_t ws_bytes) {
    (void)comm; (void)in_dtypes; (void)out_dtype; (void)d_out_all;
    if (nranks != NR || (unsigned)rank >= (unsigned)NR || n_in < 2 || ws_bytes < WS_DATA ||
        in_sizes[0] != MDIM * KLOC || in_sizes[1] != KLOC * NDIM || out_size != MLOC * NDIM) {
        fprintf(stderr, "kernel_launch: unexpected problem shape\n");
        return;
    }
    Params p = {};
    const int g4 = rank & 4, gi = rank & 3;
    const char* x = (const char*)d_in[0];
    const size_t own = (size_t)MLOC * KLOC * 2;
    p.w = (const char*)d_in[1];
    p.out = (char*)d_out;
    p.me = (char*)d_ws_all[rank];
    p.xg[0] = x + g4 * own;
    p.xg[1] = x + (g4 ^ 4) * own;
    for (int q = 0; q < 4; ++q) {
        const int o = (gi + q) & 3;
        p.xl[0][q] = x + (g4 + o) * own;
        p.xl[1][q] = x + ((g4 ^ 4) + o) * own;
        p.dst[0][q] = (char*)d_ws_all[g4 + o] + AOFF + (size_t)(q ? 4 - q : 1) * 3 * ABUF;
        p.dst[1][q] = (char*)d_ws_all[(g4 ^ 4) + o] + COFF + (size_t)gi * 3 * CBUF + PBO;
        p.crs[q] = (char*)d_ws_all[(g4 ^ 4) + q] + COFF + (size_t)gi * 3 * CBUF;
        p.park[q] = (unsigned)o * 3 * CBUF;
    }
    p.tch[0] = p.me + COFF + 6 * CBUF; p.tch[1] = p.me + COFF + 12 * CBUF - 64; p.tch[2] = p.me + WS_DATA - 64;
    for (int q = 1; q < 4; ++q) p.tch[2 + q] = p.dst[0][q];
    for (int q = 0; q < 4; ++q) p.tch[6 + q] = p.crs[q];
    p.tch[10] = p.out; p.tch[11] = p.out + (size_t)MLOC * NDIM * 2 - 64;
    const char* order = "XXWWXXWWXWXWXWXWXWXXXXWWWWXWXWXWWWWWWWWWWWWWWWWW";
    unsigned nx = 0, nw = 0, pos = 0;
    for (const char* s = order; *s; ++s, ++pos) {
        const unsigned t = T_FIRST + ((pos * UNIT16) >> 4);
        if (*s == 'X') { p.tx[nx] = t | ((t + (nx < NA ? LEAD_FAST : LEAD_LAZY)) << 16); ++nx; }
        else { p.tw[nw] = t | ((t + (nw < NFAST ? LEAD_FAST : LEAD_LAZY)) << 16); ++nw; }
    }
    for (unsigned xc = 0; xc < 8; ++xc) {
        unsigned ra = 0;
        for (unsigned k = 0; k < 4; ++k) ra += (xc >> 1) + 4 * k < NA;
        p.dt[xc] = PUSH16 * NA * 16 / ra;
        p.dt[8 + xc] = PULL16 * NB * 16 / (4 - ra);
    }
    gemm_rs_kernel<<<256, 512, 0, stream>>>(p);
    HIP_CHECK(hipGetLastError());
}
```
